```python
import numpy as np

kernel_name = "gpu_ag_gemm_m4096_k4096_n4096_w8a8_dqsilu"


ABSMAX_RTOL = 0.001

COMPUTE_DTYPE = "int8"
EPILOGUE = "dequant_silu"
IN_QUANT = "int8"
OUT_QUANT = None
IN_DTYPES = {"x": "int8", "w_mat": "int8",
             "scale_x": "float32", "scale_w": "float32"}
WORLD_SIZE = 8

def setup_inputs():

    x = np.random.randint(-127, 128, size=(4096, 4096)).astype(np.int8)
    w_mat = np.random.randint(-127, 128, size=(4096, 4096)).astype(np.int8)
    scale_x = (0.02 * (1.0 + np.random.rand(1))).astype(np.float32)
    scale_w = (0.02 * (1.0 + np.random.rand(1))).astype(np.float32)
    return {"x": x, "w_mat": w_mat, "scale_x": scale_x, "scale_w": scale_w}

def reference(x, w_mat, scale_x, scale_w):
    acc = x.astype(np.float64) @ w_mat.astype(np.float64)
    y = (acc.astype(np.float32) * (scale_x * scale_w)).astype(np.float32)
    return (y / (1.0 + np.exp(-np.clip(y, -60.0, 60.0)))).astype(np.float32)
```

```cpp
#include <hip/hip_runtime.h>
#include <rccl/rccl.h>
#include <cstdio>
#include <cstdint>

#define HIP_CHECK(expr) do { hipError_t _e = (expr); if (_e != hipSuccess) { \
    fprintf(stderr, "HIP error %s at %s:%d\n", hipGetErrorString(_e), __FILE__, __LINE__); \
    return; } } while (0)

typedef int      v4i __attribute__((ext_vector_type(4)));
typedef unsigned v4u __attribute__((ext_vector_type(4)));
typedef unsigned v2u __attribute__((ext_vector_type(2)));

constexpr int NR = 8;
constexpr int MS = 512;
constexpr int KD = 4096;
constexpr int NS = 512;
constexpr int NT = MS / 16;
constexpr int SLICE = 16 * 512;
constexpr int SHARD = MS * KD;
constexpr int NPAR = 2;
constexpr int TPW = NT / NPAR;
#define NPW 2
constexpr int NPUSH = NR - 1;
constexpr int NIDX = NPUSH + 8 * NPAR;
constexpr int GRID = 8 * NIDX;

constexpr size_t OFF_CNT  = 0;
constexpr size_t OFF_TMO  = 60 * 1024;
constexpr size_t OFF_FLAG = 64 * 1024;
constexpr size_t OFF_STAMP = 256 * 1024;
constexpr size_t OFF_DATA = 1u << 20;
constexpr size_t WS_NEED  = OFF_DATA + 2ull * NR * SHARD;
constexpr unsigned long long DEADLINE_TICKS = 50000000ull;
constexpr int RW3 = 0x00020000;
constexpr int AUX_SYS = 17;

constexpr int LDS_ACC = TPW * 1024 * 4;
constexpr int LDS_BYTES = LDS_ACC + 256;

struct Ptrs { char* ws[NR]; };

__device__ __forceinline__ void waitvm(int n) {
#define WV(k) if (n == k) asm volatile("s_waitcnt vmcnt(" #k ")" ::: "memory");
  WV(0) WV(1) WV(8) WV(9) WV(16) WV(17) WV(24) WV(25)
#undef WV
}

#define STAMP(slot) do { } while (0)

__global__ void __launch_bounds__(512, 2)
ag_gemm_kernel(const int8_t* __restrict__ x, const int8_t* __restrict__ wm,
               const float* __restrict__ sxp, const float* __restrict__ swp,
               float* __restrict__ out, Ptrs P, int rank) {
  extern __shared__ __attribute__((aligned(16))) int lds[];
  const int tid = threadIdx.x;
  const int lane = tid & 63;
  const int wave = __builtin_amdgcn_readfirstlane(tid >> 6);
  const int xcd = blockIdx.x & 7;
  const int idx = blockIdx.x >> 3;
  char* const myws = P.ws[rank];
  STAMP(0);

  if (idx < NPUSH) {
    if (wave >= NPW) return;
    unsigned* cntp = (unsigned*)(myws + OFF_CNT + (size_t)blockIdx.x * 128) + wave;
    const unsigned prev = __hip_atomic_load(cntp, __ATOMIC_RELAXED, __HIP_MEMORY_SCOPE_AGENT);
    const unsigned peer = (unsigned)(rank + 1 + idx) & (NR - 1);
    const __amdgpu_buffer_rsrc_t rs = __builtin_amdgcn_make_buffer_rsrc((void*)x, 0, SHARD, RW3);
    constexpr int STRIDE = 8 * NPW;
    constexpr int L = NT * 8 / STRIDE;
    const int gw = xcd * NPW + wave;
    const int lo = (lane & 15) * KD + (lane >> 4) * 16;
    v4u buf[2][8];
#define SRC_OFF(n) ((((n) >> 3) * 16) * KD + ((n) & 7) * 512)
#pragma unroll
    for (int s = 0; s < 8; ++s) buf[0][s] = __builtin_amdgcn_raw_buffer_load_b128(rs, lo + s * 64, SRC_OFF(gw), 0);
#pragma unroll
    for (int s = 0; s < 8; ++s) buf[1][s] = __builtin_amdgcn_raw_buffer_load_b128(rs, lo + s * 64, SRC_OFF(gw + STRIDE), 0);
    const unsigned call = __builtin_amdgcn_readfirstlane(prev) + 1u;
    char* const pws = P.ws[peer];
    const __amdgpu_buffer_rsrc_t rd = __builtin_amdgcn_make_buffer_rsrc(
        (void*)(pws + OFF_DATA + (size_t)(call & 1u) * NR * SHARD + (size_t)rank * SHARD), 0, SHARD, RW3);
    unsigned* const flags = (unsigned*)(pws + OFF_FLAG) + rank * (8 * NT);
    STAMP(1);
#pragma unroll
    for (int j = 0; j < L; ++j) {
      const int n = gw + j * STRIDE;
#pragma unroll
      for (int s = 0; s < 8; ++s)
        __builtin_amdgcn_raw_buffer_store_b128(buf[j & 1][s], rd, lane * 16 + s * 1024, n * SLICE, AUX_SYS);
      if (j + 2 < L) {
#pragma unroll
        for (int s = 0; s < 8; ++s)
          buf[j & 1][s] = __builtin_amdgcn_raw_buffer_load_b128(rs, lo + s * 64, SRC_OFF(n + 2 * STRIDE), 0);
      }
      if (j >= 1) {
        waitvm((j + 1 < L ? 8 : 0) + (j >= 2 ? 1 : 0) + 8 + (j + 2 < L ? 8 : 0));
        const int m = n - STRIDE;
        if (lane == 0)
          __hip_atomic_store(flags + (m & 7) * NT + (m >> 3), call, __ATOMIC_RELAXED, __HIP_MEMORY_SCOPE_SYSTEM);
      }
    }
    waitvm(0);
    {
      const int m = gw + (L - 1) * STRIDE;
      if (lane == 0) {
        __hip_atomic_store(flags + (m & 7) * NT + (m >> 3), call, __ATOMIC_RELAXED, __HIP_MEMORY_SCOPE_SYSTEM);
        __hip_atomic_store(cntp, call, __ATOMIC_RELAXED, __HIP_MEMORY_SCOPE_AGENT);
      }
    }
    STAMP(2);
    return;
  }

  const int ci = idx - NPUSH;
  const int slab = ci & 7;
  const int par = ci >> 3;
  const unsigned src = (unsigned)(rank + xcd) & (NR - 1);
  const bool remote = (xcd != 0);
  unsigned* cntp = (unsigned*)(myws + OFF_CNT + (size_t)blockIdx.x * 128) + wave;
  const unsigned prev = __hip_atomic_load(cntp, __ATOMIC_RELAXED, __HIP_MEMORY_SCOPE_AGENT);
  int* const lacc = lds;
  int* const ldone = lds + TPW * 1024;
  {
    v4i z = {0, 0, 0, 0};
#pragma unroll
    for (int i = 0; i < LDS_ACC / 16 / 512; ++i) ((v4i*)lacc)[i * 512 + tid] = z;
    if (tid < 64) ldone[tid] = 0;
  }
  const float scale = sxp[0] * swp[0];

  v4i Bf[8][4];
  {
    const __amdgpu_buffer_rsrc_t rw = __builtin_amdgcn_make_buffer_rsrc((void*)wm, 0, KD * NS, RW3);
    const int vo = ((lane >> 4) * 16) * NS + slab * 64 + (lane & 15) * 4;
    const int so = wave * 512 * NS;
#pragma unroll
    for (int s = 0; s < 8; ++s) {
      unsigned r[16];
#pragma unroll
      for (int j = 0; j < 16; ++j) r[j] = __builtin_amdgcn_raw_buffer_load_b32(rw, vo + (j & 7) * NS, so + (s * 64 + (j & 8)) * NS, 0);
#pragma unroll
      for (int q = 0; q < 4; ++q) {
        const unsigned t0 = __builtin_amdgcn_perm(r[4 * q + 1], r[4 * q + 0], 0x05010400u);
        const unsigned t1 = __builtin_amdgcn_perm(r[4 * q + 1], r[4 * q + 0], 0x07030602u);
        const unsigned t2 = __builtin_amdgcn_perm(r[4 * q + 3], r[4 * q + 2], 0x05010400u);
        const unsigned t3 = __builtin_amdgcn_perm(r[4 * q + 3], r[4 * q + 2], 0x07030602u);
        Bf[s][0][q] = (int)__builtin_amdgcn_perm(t2, t0, 0x05040100u);
        Bf[s][1][q] = (int)__builtin_amdgcn_perm(t2, t0, 0x07060302u);
        Bf[s][2][q] = (int)__builtin_amdgcn_perm(t3, t1, 0x05040100u);
        Bf[s][3][q] = (int)__builtin_amdgcn_perm(t3, t1, 0x07060302u);
      }
    }
  }
  const unsigned call = __builtin_amdgcn_readfirstlane(prev) + 1u;
  if (lane == 0) __hip_atomic_store(cntp, call, __ATOMIC_RELAXED, __HIP_MEMORY_SCOPE_AGENT);
  __syncthreads();
  STAMP(1);

  const __amdgpu_buffer_rsrc_t ra = remote
      ? __builtin_amdgcn_make_buffer_rsrc((void*)(myws + OFF_DATA + (size_t)(call & 1u) * NR * SHARD + (size_t)src * SHARD), 0, SHARD, RW3)
      : __builtin_amdgcn_make_buffer_rsrc((void*)x, 0, SHARD, RW3);
  const int a_lo = remote ? lane * 16 : (lane & 15) * KD + (lane >> 4) * 16;
  const int a_ss = remote ? 1024 : 64;
  const int a_wo = remote ? wave * SLICE : wave * 512;
  const unsigned* const flags = (const unsigned*)(myws + OFF_FLAG) + (src * 8 + wave) * NT + (lane & 31);
  unsigned* const tmo = (unsigned*)(myws + OFF_TMO);
  const __amdgpu_buffer_rsrc_t ro = __builtin_amdgcn_make_buffer_rsrc((void*)out, 0, NR * MS * NS * 4, RW3);

  const int e_row = 2 * wave + (lane >> 5);
  const int e_l0 = ((2 * (lane & 1)) * 4 + (e_row & 3)) * 64 + (e_row >> 2) * 16 + ((lane & 31) >> 1);
  const int e_go = ((int)(src * MS + e_row) * NS + slab * 64 + 2 * (lane & 31)) * 4;

  unsigned ready = remote ? 0u : 0xffffffffu;
  bool dead = false;
  int e = 0;

  auto epilogue = [&](int i) {
    const int a0 = lacc[i * 1024 + e_l0];
    const int a1 = lacc[i * 1024 + e_l0 + 256];
    const int bad = ldone[TPW];
    const float y0 = (float)a0 * scale, y1 = (float)a1 * scale;
    float o0 = y0 * __builtin_amdgcn_rcpf(1.0f + __expf(-fminf(fmaxf(y0, -60.0f), 60.0f)));
    float o1 = y1 * __builtin_amdgcn_rcpf(1.0f + __expf(-fminf(fmaxf(y1, -60.0f), 60.0f)));
    if (bad) { o0 = __builtin_nanf(""); o1 = o0; }
    v2u o = {__builtin_bit_cast(unsigned, o0), __builtin_bit_cast(unsigned, o1)};
    __builtin_amdgcn_raw_buffer_store_b64(o, ro, e_go, (i * NPAR + par) * 16 * NS * 4, 0);
  };

  for (int i = 0; i < TPW; ++i) {
    const int t = i * NPAR + par;
    if (!((ready >> t) & 1u)) {
      const unsigned long long t0 = __builtin_amdgcn_s_memrealtime();
      for (;;) {
        const unsigned f = __hip_atomic_load(flags, __ATOMIC_RELAXED, __HIP_MEMORY_SCOPE_SYSTEM);
        ready = (unsigned)__builtin_amdgcn_ballot_w64(f >= call);
        if ((ready >> t) & 1u) break;
        if (__builtin_amdgcn_readfirstlane(__hip_atomic_load(tmo, __ATOMIC_RELAXED, __HIP_MEMORY_SCOPE_AGENT)) == call ||
            __builtin_amdgcn_s_memrealtime() - t0 > DEADLINE_TICKS) {
          if (lane == 0) {
            __hip_atomic_store(tmo, call, __ATOMIC_RELAXED, __HIP_MEMORY_SCOPE_AGENT);
            __hip_atomic_store(&ldone[TPW], 1, __ATOMIC_RELAXED, __HIP_MEMORY_SCOPE_WORKGROUP);
          }
          dead = true;
          ready = 0xffffffffu;
          break;
        }
        __builtin_amdgcn_s_sleep(2);
      }
      asm volatile("" ::: "memory");
    }
    if (i == TPW - 1) STAMP(2);
    if (!dead) {
      v4u A[8];
#pragma unroll
      for (int s = 0; s < 8; ++s)
        A[s] = __builtin_amdgcn_raw_buffer_load_b128(ra, a_lo + s * a_ss, a_wo + t * (16 * KD), AUX_SYS);
      v4i acc[4];
#pragma unroll
      for (int n = 0; n < 4; ++n) acc[n] = (v4i){0, 0, 0, 0};
#pragma unroll
      for (int s = 0; s < 8; ++s)
#pragma unroll
        for (int n = 0; n < 4; ++n)
          acc[n] = __builtin_amdgcn_mfma_i32_16x16x64_i8(__builtin_bit_cast(v4i, A[s]), Bf[s][n], acc[n], 0, 0, 0);
#pragma unroll
      for (int n = 0; n < 4; ++n)
#pragma unroll
        for (int r = 0; r < 4; ++r)
          __hip_atomic_fetch_add(&lacc[i * 1024 + (n * 4 + r) * 64 + lane], acc[n][r], __ATOMIC_RELAXED, __HIP_MEMORY_SCOPE_WORKGROUP);
    }
    asm volatile("" ::: "memory");
    if (lane == 0) __hip_atomic_fetch_add(&ldone[i], 1, __ATOMIC_RELAXED, __HIP_MEMORY_SCOPE_WORKGROUP);
    asm volatile("" ::: "memory");
    if (i == TPW - 1) STAMP(3);
    while (e <= i &&
           __builtin_amdgcn_readfirstlane(__hip_atomic_load(&ldone[e], __ATOMIC_RELAXED, __HIP_MEMORY_SCOPE_WORKGROUP)) == 8) {
      asm volatile("" ::: "memory");
      epilogue(e);
      ++e;
    }
  }
  while (e < TPW) {
    while (__builtin_amdgcn_readfirstlane(__hip_atomic_load(&ldone[e], __ATOMIC_RELAXED, __HIP_MEMORY_SCOPE_WORKGROUP)) != 8)
      __builtin_amdgcn_s_sleep(1);
    asm volatile("" ::: "memory");
    epilogue(e);
    ++e;
  }
  STAMP(4);
}

extern "C" void kernel_launch(int rank, int nranks, ncclComm_t comm,
                              hipStream_t stream,
                              void* const* d_in, const int* in_sizes, int n_in,
                              void* d_out, int out_size,
                              const int* in_dtypes, int out_dtype,
                              void* const* d_out_all,
                              void* const* d_ws_all, size_t ws_bytes) {
  if (nranks != NR || (unsigned)rank >= (unsigned)NR || n_in < 4 || ws_bytes < WS_NEED ||
      in_sizes[0] != MS * KD || in_sizes[1] != KD * NS || out_size != NR * MS * NS) {
    fprintf(stderr, "kernel_launch: unexpected problem shape\n");
    return;
  }
  Ptrs P;
  for (int p = 0; p < NR; ++p) P.ws[p] = (char*)d_ws_all[p];
  HIP_CHECK(hipFuncSetAttribute((const void*)ag_gemm_kernel, hipFuncAttributeMaxDynamicSharedMemorySize, LDS_BYTES));
  hipLaunchKernelGGL(ag_gemm_kernel, dim3(GRID), dim3(512), LDS_BYTES, stream,
                     (const int8_t*)d_in[0], (const int8_t*)d_in[1], (const float*)d_in[2], (const float*)d_in[3],
                     (float*)d_out, P, rank);
  HIP_CHECK(hipGetLastError());
}
```

```cpp
#define NPAR 3
#define PWG 8
#define LTICK 4
#define WLOOP 17
#define WEND 8
#include <hip/hip_runtime.h>
#include <rccl/rccl.h>
#include <cstdio>
#include <cstdint>

#define HIP_CHECK(expr) do { hipError_t _e = (expr); if (_e != hipSuccess) { \
    fprintf(stderr, "HIP error %s at %s:%d\n", hipGetErrorString(_e), __FILE__, __LINE__); \
    return; } } while (0)

typedef int      v4i __attribute__((ext_vector_type(4)));
typedef unsigned v4u __attribute__((ext_vector_type(4)));

constexpr int NR = 8;
constexpr int MS = 512;
constexpr int KD = 4096;
constexpr int NS = 512;
constexpr int NT = MS / 16;
constexpr int SLICE = 16 * 512;
constexpr int NSL = NT * 8;
constexpr int SHARD = MS * KD;
constexpr int TPW = (NT + NPAR - 1) / NPAR;
#define NPW 1
#define NEAR 1
#define AUX1 17
#define GLEAD 0
#define GOFF (SLICE - 4)
#define ZDELAY 800
#define SLP 1
#undef NPW
#define NPW (8 / PWG)
constexpr int NW = 8;
constexpr int NPSLOT = PWG > 8 ? PWG : 8;
constexpr int NCONS = 8 * NPAR;
constexpr int NIDX = 32;
constexpr int GRID = 8 * NIDX;
constexpr int NZCOL = NIDX - NPSLOT - NCONS;
#define NZU 8
constexpr int NZ = PWG ? NZU : 8 * NZCOL + 8;

constexpr size_t OFF_CNT  = 0;
constexpr size_t OFF_TMO  = 60 * 1024;
constexpr size_t OFF_TICK = 64 * 1024;
constexpr size_t OFF_FLAG = 72 * 1024;
constexpr size_t OFF_STAMP = 256 * 1024;
constexpr size_t OFF_DATA = 1u << 20;
constexpr size_t WS_NEED  = OFF_DATA + 3ull * NR * SHARD;
constexpr unsigned long long DEADLINE_TICKS = 50000000ull;
constexpr int RW3 = 0x00020000;
constexpr int AUX_SYS = 17;
constexpr int AUX_DEV = 16;
constexpr int OOB = 0x7ffffff0;

constexpr int LDS_ACC = TPW * 1024 * 4;
constexpr int LDS_BYTES = LDS_ACC + 256;

#define CLOB "v64", "v65", "v66", "v67", "v68", "v69", "v70", "v71", "v72", "v73", "v74", "v75", "v76", "v77", "v78", "v79", "v80", "v81", "v82", "v83", "v84", "v85", "v86", "v87", "v88", "v89", "v90", "v91", "v92", "v93", "v94", "v95", "v96", "v97", "v98", "v99", "v100", "v101", "v102", "v103", "v104", "v105", "v106", "v107", "v108", "v109", "v110", "v111", "v112", "v113", "v114", "v115", "v116", "v117", "v118", "v119", "v120", "v121", "v122", "v123", "v124", "v125", "v126", "v127", "v128"
struct Ptrs { char* ws[NR]; };

#define STAMP(slot) do { } while (0)
#define STAMPV(slot, v) do { } while (0)

__device__ __forceinline__ unsigned umin4(v4u a) {
  return __builtin_elementwise_min(__builtin_elementwise_min(a.x, a.y), __builtin_elementwise_min(a.z, a.w));
}

__global__ void __launch_bounds__(512, 2)
ag_gemm_kernel(const int8_t* __restrict__ x, const int8_t* __restrict__ wm,
               const float* __restrict__ sxp, const float* __restrict__ swp,
               float* __restrict__ out, Ptrs P, int rank, unsigned peers) {
  extern __shared__ __attribute__((aligned(16))) int lds[];
  const int tid = threadIdx.x;
  const int lane = tid & 63;
  const int wave = __builtin_amdgcn_readfirstlane(tid >> 6);
  const int xcd = blockIdx.x & 7;
  const int idx = blockIdx.x >> 3;
  char* const myws = P.ws[rank];
  STAMP(0);
  unsigned* const cntp = (unsigned*)(myws + OFF_CNT + (size_t)blockIdx.x * 128) + wave;

  int plink = -1, psub = 0, zid = -1;
  if (PWG) {
    if (idx < NPSLOT || idx >= NPSLOT + NCONS) {
      if (xcd != 3 && idx < PWG) { plink = xcd < 3 ? xcd : xcd - 1; psub = idx; }
      else {
        zid = (NPSLOT - 1 - idx) * 8 + xcd;
        if (PWG >= 8) zid = xcd == 3 ? idx : -1;
        if (idx >= NPSLOT || zid < 0 || zid >= NZ) return;
      }
    }
  } else if (idx < NPSLOT) {
    if (NEAR) {
      if (xcd < 4) { if (idx < 6) { plink = idx % 3; psub = xcd * 2 + idx / 3; } else zid = xcd * 2 + (idx - 6); }
      else { plink = 3 + (idx & 3); psub = (xcd - 4) * 2 + (idx >> 2); }
    } else {
      if (idx < 7) { plink = idx; psub = xcd; } else zid = xcd;
    }
  } else if (idx >= NPSLOT + NCONS) {
    zid = 8 + (idx - NPSLOT - NCONS) * 8 + xcd;
  }

  if (plink >= 0) {
    if (wave >= NPW) return;
    {
      const unsigned peer = __builtin_amdgcn_readfirstlane((peers >> (4 * plink)) & 7u);
      char* const pws = P.ws[peer];
      const __amdgpu_buffer_rsrc_t rs = __builtin_amdgcn_make_buffer_rsrc((void*)x, 0, SHARD, RW3);
      const __amdgpu_buffer_rsrc_t rd = __builtin_amdgcn_make_buffer_rsrc((void*)(pws + OFF_DATA), 0, 3 * NR * SHARD, RW3);
      const __amdgpu_buffer_rsrc_t rt = __builtin_amdgcn_make_buffer_rsrc((void*)(myws + OFF_TICK), 0, 3 * NR * 128, RW3);
      const int g = __builtin_amdgcn_readfirstlane(psub * NPW + wave);
      const int lo = (lane & 15) * KD + (lane >> 4) * 16;
      const int d0 = lane * 16, d1 = lane * 16 + 4096;
      const int l0 = lane == 0 ? plink * 128 : OOB;
      unsigned call, dbase, oA, oB, t, u, to;
#define LD1(R, O, SO) "buffer_load_dwordx4 v[" #R ":" #R "+3], %[lo], %[rs], %[" SO "] offen offset:" #O "\n"
#define LDA(SO) LD1(64,0,SO) LD1(68,64,SO) LD1(72,128,SO) LD1(76,192,SO) LD1(80,256,SO) LD1(84,320,SO) LD1(88,384,SO) LD1(92,448,SO)
#define LDB(SO) LD1(96,0,SO) LD1(100,64,SO) LD1(104,128,SO) LD1(108,192,SO) LD1(112,256,SO) LD1(116,320,SO) LD1(120,384,SO) LD1(124,448,SO)
#define ST1(R, W, D, O, SO) "s_waitcnt vmcnt(" W ")\n" \
      "v_xor_b32 v[" #R "], %[x80], v[" #R "]\n" "v_xor_b32 v[" #R "+1], %[x80], v[" #R "+1]\n" \
      "v_xor_b32 v[" #R "+2], %[x80], v[" #R "+2]\n" "v_xor_b32 v[" #R "+3], %[x80], v[" #R "+3]\n" \
      "buffer_store_dwordx4 v[" #R ":" #R "+3], %[" #D "], %[rd], %[" SO "] offen offset:" #O " sc0 sc1\n"
#define STA(W) ST1(64,W,d0,0,"oA") ST1(68,W,d0,1024,"oA") ST1(72,W,d0,2048,"oA") ST1(76,W,d0,3072,"oA") \
               ST1(80,W,d1,0,"oA") ST1(84,W,d1,1024,"oA") ST1(88,W,d1,2048,"oA") ST1(92,W,d1,3072,"oA")
#define STB(W) ST1(96,W,d0,0,"oB") ST1(100,W,d0,1024,"oB") ST1(104,W,d0,2048,"oB") ST1(108,W,d0,3072,"oB") \
               ST1(112,W,d1,0,"oB") ST1(116,W,d1,1024,"oB") ST1(120,W,d1,2048,"oB") ST1(124,W,d1,3072,"oB")
#define OFFS(O) "s_lshl_b32 %[u], %[t], 13\n" "s_add_u32 %[" O "], %[dbase], %[u]\n" \
      "s_and_b32 %[u], %[u], 0xffff0000\n" "s_and_b32 %[t], %[t], 7\n" "s_lshl_b32 %[t], %[t], 9\n" "s_or_b32 %[t], %[t], %[u]\n"
#define SRCO "s_lshl_b32 %[u], %[t], 13\n" \
      "s_and_b32 %[u], %[u], 0xffff0000\n" "s_and_b32 %[t], %[t], 7\n" "s_lshl_b32 %[t], %[t], 9\n" "s_or_b32 %[t], %[t], %[u]\n"
#define STR2(x) #x
#define STR(x) STR2(x)
#define TICKET(R) "buffer_load_dword v" #R ", %[l0], %[rt], 0 offen offset:4000\n"
#define GETT(R, O, LBL) "s_add_u32 %[to], %[to], %[nw]\n" "s_mov_b32 %[t], %[to]\n" \
      "s_cmp_ge_u32 %[t], %[lim]\n" "s_cbranch_scc1 " LBL "%=\n" OFFS(O)
      asm volatile(
        "global_load_dword v128, %[cnt], off sc1\n"
        "s_mov_b32 %[t], %[g]\n" SRCO
        LDA("t")
        "s_add_u32 %[t], %[g], %[nw]\n" SRCO
        LDB("t")
        "s_waitcnt vmcnt(16)\n"
        "v_readfirstlane_b32 %[call], v128\n"
        "s_add_u32 %[call], %[call], 1\n"
        "s_mul_hi_u32 %[t], %[call], 0xaaaaaaab\n"
        "s_lshr_b32 %[t], %[t], 1\n"
        "s_mul_i32 %[t], %[t], 3\n"
        "s_sub_u32 %[t], %[call], %[t]\n"
        "s_lshl_b32 %[to], %[t], 10\n"
        "s_lshl_b32 %[t], %[t], 24\n"
        "s_add_u32 %[dbase], %[t], %[rk]\n"
        "s_lshl_b32 %[t], %[g], 13\n"
        "s_add_u32 %[oA], %[dbase], %[t]\n"
        "s_lshl_b32 %[t], %[nw], 13\n"
        "s_add_u32 %[oB], %[oA], %[t]\n"
        TICKET(128) STA("16")
        "s_add_u32 %[t], %[g], %[nw]\n" "s_add_u32 %[t], %[t], %[nw]\n"
        "s_mov_b32 %[to], %[t]\n"
        OFFS("oA") LDA("t")
        "L_loop_%=:\n"
        TICKET(129) STB(STR(WLOOP)) GETT(128, "oB", "L_endA_") LDB("t")
        TICKET(128) STA(STR(WLOOP)) GETT(129, "oA", "L_endB_") LDA("t")
        "s_branch L_loop_%=\n"
        "L_endA_%=:\n"
        STA(STR(WEND))
        "s_branch L_done_%=\n"
        "L_endB_%=:\n"
        STB(STR(WEND))
        "L_done_%=:\n"
        : [call] "=&s"(call), [dbase] "=&s"(dbase), [oA] "=&s"(oA), [oB] "=&s"(oB), [t] "=&s"(t), [u] "=&s"(u), [to] "=&s"(to)
        : [lo] "v"(lo), [d0] "v"(d0), [d1] "v"(d1), [l0] "v"(l0), [rs] "s"(rs), [rd] "s"(rd), [rt] "s"(rt), [cnt] "v"(cntp),
          [g] "s"(g), [rk] "s"(rank * SHARD), [x80] "s"(0x80808080u), [lim] "s"(NSL), [nw] "s"(NW), [nw3] "s"(3 * NW)
        : "memory", "scc", CLOB, "v129");
      STAMP(2);
      asm volatile("s_waitcnt vmcnt(0)" ::: "memory");
      if (lane == 0) {
        __hip_atomic_store((unsigned*)(pws + OFF_FLAG + (size_t)rank * 128) + g, call, __ATOMIC_RELAXED, __HIP_MEMORY_SCOPE_SYSTEM);
        __hip_atomic_store(cntp, call, __ATOMIC_RELAXED, __HIP_MEMORY_SCOPE_AGENT);
      }
      STAMP(3);
      return;
    }
    const unsigned prev = __hip_atomic_load(cntp, __ATOMIC_RELAXED, __HIP_MEMORY_SCOPE_AGENT);
    const unsigned peer = (peers >> (4 * plink)) & 7u;
    const __amdgpu_buffer_rsrc_t rs = __builtin_amdgcn_make_buffer_rsrc((void*)x, 0, SHARD, RW3);
    const int g = psub * NPW + wave;
    const int lo = (lane & 15) * KD + (lane >> 4) * 16;
    v4u b0[8], b1[8];
#define SRC_OFF(n) ((((n) >> 3) * 16) * KD + ((n) & 7) * 512)
#pragma unroll
    for (int s = 0; s < 8; ++s) b0[s] = __builtin_amdgcn_raw_buffer_load_b128(rs, lo + s * 64, SRC_OFF(g), 0);
#pragma unroll
    for (int s = 0; s < 8; ++s) b1[s] = __builtin_amdgcn_raw_buffer_load_b128(rs, lo + s * 64, SRC_OFF(g + NW), 0);
    const unsigned call = __builtin_amdgcn_readfirstlane(prev) + 1u;
    const unsigned cb = call % 3u;
    char* const pws = P.ws[peer];
    const __amdgpu_buffer_rsrc_t rd = __builtin_amdgcn_make_buffer_rsrc(
        (void*)(pws + OFF_DATA + ((size_t)cb * NR + (size_t)rank) * SHARD), 0, SHARD, RW3);
    STAMP(1);
    int nsent = NSL / NW;
#pragma unroll
    for (int j = 0; j < NSL / NW; ++j) {
      const int n = g + j * NW;
      v4u (&bb)[8] = (j & 1) ? b1 : b0;
#pragma unroll
      for (int s = 0; s < 8; ++s)
        __builtin_amdgcn_raw_buffer_store_b128(bb[s] ^ 0x80808080u, rd, lane * 16 + s * 1024, n * SLICE, AUX_SYS);
      if (j + 2 < NSL / NW) {
#pragma unroll
        for (int s = 0; s < 8; ++s) bb[s] = __builtin_amdgcn_raw_buffer_load_b128(rs, lo + s * 64, SRC_OFF(n + 2 * NW), 0);
      }
    }
    STAMP(2);
    asm volatile("s_waitcnt vmcnt(0)" ::: "memory");
    if (lane == 0) {
      __hip_atomic_store((unsigned*)(pws + OFF_FLAG + (size_t)rank * 128) + g, call, __ATOMIC_RELAXED, __HIP_MEMORY_SCOPE_SYSTEM);
      __hip_atomic_store(cntp, call, __ATOMIC_RELAXED, __HIP_MEMORY_SCOPE_AGENT);
    }
    STAMP(3);
    STAMPV(8, nsent);
    return;
  }

  if (zid >= 0) {
    const unsigned long long t0 = __builtin_amdgcn_s_memrealtime();
    const unsigned prev = __hip_atomic_load(cntp, __ATOMIC_RELAXED, __HIP_MEMORY_SCOPE_AGENT);
    const unsigned call = __builtin_amdgcn_readfirstlane(prev) + 1u;
    if (lane == 0) __hip_atomic_store(cntp, call, __ATOMIC_RELAXED, __HIP_MEMORY_SCOPE_AGENT);
    const unsigned zb = (call + 2u) % 3u;
    if (zid == 0 && tid < NR)
      __hip_atomic_store((unsigned*)(myws + OFF_TICK + ((size_t)zb * NR + tid) * 128), 0u, __ATOMIC_RELAXED, __HIP_MEMORY_SCOPE_AGENT);
    while (__builtin_amdgcn_s_memrealtime() - t0 < ZDELAY) __builtin_amdgcn_s_sleep(8);
    const __amdgpu_buffer_rsrc_t rz = __builtin_amdgcn_make_buffer_rsrc(
        (void*)(myws + OFF_DATA + (size_t)zb * NR * SHARD), 0, NR * SHARD, RW3);
    const v4u z = {0u, 0u, 0u, 0u};
    for (int u = zid; u < (NR - 1) * NSL; u += NZ) {
      const int v = u + ((u >> 8) >= rank ? NSL : 0);
      __builtin_amdgcn_raw_buffer_store_b128(z, rz, tid * 16, v * SLICE, AUX_DEV);
    }
    STAMP(1);
    return;
  }

  const int ci = idx - NPSLOT;
  const int slab = ci & 7;
  const int par = ci >> 3;
  const int ntile = (NT - par + NPAR - 1) / NPAR;
  const unsigned src = (unsigned)(rank + xcd) & (NR - 1);
  const bool remote = (xcd != 0);
  const unsigned prev = __hip_atomic_load(cntp, __ATOMIC_RELAXED, __HIP_MEMORY_SCOPE_AGENT);
  int* const lacc = lds;
  int* const ldone = lds + TPW * 1024;
  {
    v4i z = {0, 0, 0, 0};
#pragma unroll
    for (int i = 0; i < (LDS_ACC / 16 + 511) / 512; ++i)
      if (i * 512 + tid < LDS_ACC / 16) ((v4i*)lacc)[i * 512 + tid] = z;
    if (tid < 64) ldone[tid] = 0;
  }
  const float scale = sxp[0] * swp[0];

  v4i Bf[8][4];
  {
    const __amdgpu_buffer_rsrc_t rw = __builtin_amdgcn_make_buffer_rsrc((void*)wm, 0, KD * NS, RW3);
    const int vo = ((lane >> 4) * 16) * NS + slab * 64 + (lane & 15) * 4;
    const int so = wave * 512 * NS;
#pragma unroll
    for (int s = 0; s < 8; ++s) {
      unsigned r[16];
#pragma unroll
      for (int j = 0; j < 16; ++j) r[j] = __builtin_amdgcn_raw_buffer_load_b32(rw, vo + (j & 7) * NS, so + (s * 64 + (j & 8)) * NS, 0);
#pragma unroll
      for (int q = 0; q < 4; ++q) {
        const unsigned t0 = __builtin_amdgcn_perm(r[4 * q + 1], r[4 * q + 0], 0x05010400u);
        const unsigned t1 = __builtin_amdgcn_perm(r[4 * q + 1], r[4 * q + 0], 0x07030602u);
        const unsigned t2 = __builtin_amdgcn_perm(r[4 * q + 3], r[4 * q + 2], 0x05010400u);
        const unsigned t3 = __builtin_amdgcn_perm(r[4 * q + 3], r[4 * q + 2], 0x07030602u);
        Bf[s][0][q] = (int)__builtin_amdgcn_perm(t2, t0, 0x05040100u);
        Bf[s][1][q] = (int)__builtin_amdgcn_perm(t2, t0, 0x07060302u);
        Bf[s][2][q] = (int)__builtin_amdgcn_perm(t3, t1, 0x05040100u);
        Bf[s][3][q] = (int)__builtin_amdgcn_perm(t3, t1, 0x07060302u);
      }
    }
  }
  const unsigned call = __builtin_amdgcn_readfirstlane(prev) + 1u;
  if (lane == 0) __hip_atomic_store(cntp, call, __ATOMIC_RELAXED, __HIP_MEMORY_SCOPE_AGENT);
  __syncthreads();
  STAMP(1);

  const __amdgpu_buffer_rsrc_t ro = __builtin_amdgcn_make_buffer_rsrc((void*)out, 0, NR * MS * NS * 4, RW3);
  const int e_row = 2 * wave + ((lane >> 4) & 1);
  const int e_l0 = (e_row & 3) * 64 + (e_row >> 2) * 16 + (lane & 15);
  const int e_go = lane < 32 ? ((int)(src * MS + e_row) * NS + slab * 64 + 4 * (lane & 15)) * 4 : OOB;
  int e = 0;

  auto epilogue = [&](int i) {
    const int bad = ldone[63];
    v4u o;
#pragma unroll
    for (int n = 0; n < 4; ++n) {
      const float y = (float)lacc[i * 1024 + n * 256 + e_l0] * scale;
      float v = y * __builtin_amdgcn_rcpf(1.0f + __expf(-fminf(fmaxf(y, -60.0f), 60.0f)));
      if (bad) v = __builtin_nanf("");
      o[n] = __builtin_bit_cast(unsigned, v);
    }
    __builtin_amdgcn_raw_buffer_store_b128(o, ro, e_go, (i * NPAR + par) * 16 * NS * 4, AUX_DEV);
  };
  auto finish = [&](int i, const v4i (&acc)[4], bool add) {
    if (add) {
#pragma unroll
      for (int n = 0; n < 4; ++n)
#pragma unroll
        for (int r = 0; r < 4; ++r)
          __hip_atomic_fetch_add(&lacc[i * 1024 + (n * 4 + r) * 64 + lane], acc[n][r], __ATOMIC_RELAXED, __HIP_MEMORY_SCOPE_WORKGROUP);
    }
    asm volatile("" ::: "memory");
    if (lane == 0) __hip_atomic_fetch_add(&ldone[i], 1, __ATOMIC_RELAXED, __HIP_MEMORY_SCOPE_WORKGROUP);
    asm volatile("" ::: "memory");
    while (e <= i &&
           __builtin_amdgcn_readfirstlane(__hip_atomic_load(&ldone[e], __ATOMIC_RELAXED, __HIP_MEMORY_SCOPE_WORKGROUP)) == 8) {
      asm volatile("" ::: "memory");
      epilogue(e);
      ++e;
    }
  };

  if (!remote) {
    const __amdgpu_buffer_rsrc_t ra = __builtin_amdgcn_make_buffer_rsrc((void*)x, 0, SHARD, RW3);
    const int a_lo = (lane & 15) * KD + (lane >> 4) * 16 + wave * 512;
    v4u A[8];
#pragma unroll
    for (int s = 0; s < 8; ++s) A[s] = __builtin_amdgcn_raw_buffer_load_b128(ra, a_lo + s * 64, par * (16 * KD), 0);
    for (int i = 0; i < ntile; ++i) {
      const int so = i + 1 < ntile ? ((i + 1) * NPAR + par) * (16 * KD) : OOB;
      v4i acc[4];
#pragma unroll
      for (int n = 0; n < 4; ++n) acc[n] = (v4i){0, 0, 0, 0};
      v4u An[8];
#pragma unroll
      for (int s = 0; s < 8; ++s) An[s] = __builtin_amdgcn_raw_buffer_load_b128(ra, a_lo + s * 64, so, 0);
#pragma unroll
      for (int s = 0; s < 8; ++s)
#pragma unroll
        for (int n = 0; n < 4; ++n)
          acc[n] = __builtin_amdgcn_mfma_i32_16x16x64_i8(__builtin_bit_cast(v4i, A[s]), Bf[s][n], acc[n], 0, 0, 0);
      finish(i, acc, true);
#pragma unroll
      for (int s = 0; s < 8; ++s) A[s] = An[s];
    }
  } else {
    const unsigned cb = call % 3u;
    const __amdgpu_buffer_rsrc_t rb = __builtin_amdgcn_make_buffer_rsrc((void*)myws, 0, (int)WS_NEED, RW3);
    const int dbase = (int)OFF_DATA + (int)(cb * NR + src) * SHARD;
    const int f_lo = lane < NW ? (int)OFF_FLAG + (int)src * 128 + lane * 4 : OOB;
    constexpr unsigned long long FMASK = NW == 64 ? ~0ull : ((1ull << NW) - 1ull);
    unsigned* const tmo = (unsigned*)(myws + OFF_TMO);
    bool fin = false;
    bool dead = false;
    for (int i = 0; i < ntile; ++i) {
      const int t = i * NPAR + par;
      const int n = t * 8 + wave;
      unsigned long long t0 = 0;
      unsigned spins = 0;
      auto expired = [&]() -> bool {
        bool ex = false;
        if (spins == 0) t0 = __builtin_amdgcn_s_memrealtime();
        if ((++spins & 255u) == 0) {
          ex = __builtin_amdgcn_s_memrealtime() - t0 > DEADLINE_TICKS ||
               __builtin_amdgcn_readfirstlane(__hip_atomic_load(tmo, __ATOMIC_RELAXED, __HIP_MEMORY_SCOPE_AGENT)) == call;
          if (ex) {
            __hip_atomic_store(tmo, call, __ATOMIC_RELAXED, __HIP_MEMORY_SCOPE_AGENT);
            __hip_atomic_store(&ldone[63], 1, __ATOMIC_RELAXED, __HIP_MEMORY_SCOPE_WORKGROUP);
          }
        }
        return ex;
      };
      if (!fin && !dead) {
        const int gn = (i == ntile - 1) ? n - GLEAD : n;
        const int g_lo = lane == 63 ? dbase + gn * SLICE + GOFF : f_lo;
        for (;;) {
          const unsigned v = __builtin_amdgcn_raw_buffer_load_b32(rb, g_lo, 0, AUX_SYS);
          const unsigned long long m = __builtin_amdgcn_ballot_w64(lane == 63 ? v != 0u : v >= call);
          if ((m & FMASK) == FMASK) { fin = true; break; }
          if (m >> 63) break;
          if (expired()) { dead = true; break; }
          __builtin_amdgcn_s_sleep(SLP);
        }
        asm volatile("" ::: "memory");
      }
      if (i == ntile - 1) STAMP(2);
      v4i acc[4];
#pragma unroll
      for (int q = 0; q < 4; ++q) acc[q] = (v4i){0, 0, 0, 0};
      if (!dead) {
        const int so = dbase + n * SLICE;
        v4u A[8];
#pragma unroll
        for (int s = 0; s < 8; ++s) A[s] = __builtin_amdgcn_raw_buffer_load_b128(rb, lane * 16 + s * 1024, so, AUX1);
        unsigned miss = 0;
#pragma unroll
        for (int s = 0; s < 8; ++s) {
          if ((AUX1 == AUX_SYS && fin) || __builtin_amdgcn_ballot_w64(umin4(A[s]) == 0u) == 0ull) {
            const v4i a = __builtin_bit_cast(v4i, A[s] ^ 0x80808080u);
#pragma unroll
            for (int q = 0; q < 4; ++q) acc[q] = __builtin_amdgcn_mfma_i32_16x16x64_i8(a, Bf[s][q], acc[q], 0, 0, 0);
          } else {
            miss |= 1u << s;
          }
        }
        if (i == ntile - 1) STAMPV(10, miss);
        while (miss) {
          const unsigned v = __builtin_amdgcn_raw_buffer_load_b32(rb, f_lo, 0, AUX_SYS);
#pragma unroll
          for (int s = 0; s < 8; ++s)
            if ((miss >> s) & 1u) A[s] = __builtin_amdgcn_raw_buffer_load_b128(rb, lane * 16 + s * 1024, so, AUX_SYS);
          const bool fin_next = (__builtin_amdgcn_ballot_w64(v >= call) & FMASK) == FMASK;
#pragma unroll
          for (int s = 0; s < 8; ++s) {
            if (((miss >> s) & 1u) && (fin || __builtin_amdgcn_ballot_w64(umin4(A[s]) == 0u) == 0ull)) {
              const v4i a = __builtin_bit_cast(v4i, A[s] ^ 0x80808080u);
#pragma unroll
              for (int q = 0; q < 4; ++q) acc[q] = __builtin_amdgcn_mfma_i32_16x16x64_i8(a, Bf[s][q], acc[q], 0, 0, 0);
              miss &= ~(1u << s);
            }
          }
          fin = fin_next;
          asm volatile("" ::: "memory");
          if (miss && expired()) { dead = true; break; }
        }
      }
      if (i == ntile - 1) STAMP(3);
      finish(i, acc, !dead);
    }
  }
  while (e < ntile) {
    while (__builtin_amdgcn_readfirstlane(__hip_atomic_load(&ldone[e], __ATOMIC_RELAXED, __HIP_MEMORY_SCOPE_WORKGROUP)) != 8)
      __builtin_amdgcn_s_sleep(1);
    asm volatile("" ::: "memory");
    epilogue(e);
    ++e;
  }
  STAMP(4);
}

extern "C" void kernel_launch(int rank, int nranks, ncclComm_t comm,
                              hipStream_t stream,
                              void* const* d_in, const int* in_sizes, int n_in,
                              void* d_out, int out_size,
                              const int* in_dtypes, int out_dtype,
                              void* const* d_out_all,
                              void* const* d_ws_all, size_t ws_bytes) {
  if (nranks != NR || (unsigned)rank >= (unsigned)NR || n_in < 4 || ws_bytes < WS_NEED ||
      in_sizes[0] != MS * KD || in_sizes[1] != KD * NS || out_size != NR * MS * NS) {
    fprintf(stderr, "kernel_launch: unexpected problem shape\n");
    return;
  }
  Ptrs P;
  for (int p = 0; p < NR; ++p) P.ws[p] = (char*)d_ws_all[p];
  const unsigned G = 0x4Du;
  unsigned peers = 0; int k = 0;
  for (int pass = 0; pass < 2; ++pass)
    for (int d = 1; d < NR; ++d) {
      const int p = (rank + d) & (NR - 1);
      const bool same = ((G >> p) & 1u) == ((G >> rank) & 1u);
      if (same == (pass == 0)) peers |= (unsigned)p << (4 * k++);
    }
  HIP_CHECK(hipFuncSetAttribute((const void*)ag_gemm_kernel, hipFuncAttributeMaxDynamicSharedMemorySize, LDS_BYTES));
  hipLaunchKernelGGL(ag_gemm_kernel, dim3(GRID), dim3(512), LDS_BYTES, stream,
                     (const int8_t*)d_in[0], (const int8_t*)d_in[1], (const float*)d_in[2], (const float*)d_in[3],
                     (float*)d_out, P, rank, peers);
  HIP_CHECK(hipGetLastError());
}
```

```cpp
#include <hip/hip_runtime.h>
#include <rccl/rccl.h>
#include <cstdio>
#include <cstdint>

#define HIP_CHECK(expr) do { hipError_t _e = (expr); if (_e != hipSuccess) { \
    fprintf(stderr, "HIP error %s at %s:%d\n", hipGetErrorString(_e), __FILE__, __LINE__); \
    return; } } while (0)

typedef int      v4i __attribute__((ext_vector_type(4)));
typedef unsigned v4u __attribute__((ext_vector_type(4)));
typedef unsigned v2u __attribute__((ext_vector_type(2)));

constexpr int NR = 8;
constexpr int MS = 512;
constexpr int KD = 4096;
constexpr int NS = 512;
constexpr int NT = MS / 16;
constexpr int SLICE = 16 * 512;
constexpr int TILE = 8 * SLICE;
constexpr int SHARD = MS * KD;
#define NU 27
#define NQ 0
constexpr int NP = NT - NU - NQ;
#define PI0 4
#define PI1 6
#define PIH 4
#define GOFF (SLICE - 4)
#define GEARLY 4096
#define GEARLYH GEARLY
#define ALT 0
#define FMODE 0
#define FSTR 4
#define CLR 1
#define PI1H 5
#define FORD 1
#define CDELAY 0
#define ZDELAY 800
#define SLP 1
#define WLOOP 17
#define WEND 8
#define WINW 9
#define WINX 9
#define XMAP 0x76543210u
#define LAY 2
#define LAY1(c) (LAY & 1 ? __builtin_expect(!!(c), 1) : (c))
#define LAY2(c) (LAY & 2 ? __builtin_expect(!!(c), 1) : (c))
#define WARMC 3
#define WARMP 0
constexpr int NW = 8;
constexpr int NWG = 12;
constexpr int NPAR = NWG / 3;
constexpr int GRID = 256;
constexpr int NSLI = (NU + NQ) * 8;
constexpr int NXE = NP + NQ;
constexpr int cdiv(int a, int b) { return (a + b - 1) / b; }
constexpr int MAXI = cdiv(NT, NWG) + cdiv(NU + NQ, NPAR) + cdiv(4 * NXE, NWG);
static_assert(NU >= 8 && NP >= 0 && 4 * NXE <= 2 * NWG && 3 * NP <= 2 * NWG && MAXI <= 16, "config");

constexpr size_t OFF_CNT  = 0;
constexpr size_t OFF_TMO  = 60 * 1024;
constexpr size_t OFF_FLAG = 72 * 1024;
constexpr size_t OFF_RFLAG = 128 * 1024;
constexpr int NFW = FMODE ? 1 : ALT ? 8 : 4;
constexpr int NFL = 2 * NWG * NFW;
constexpr int RFS = 32 * 1024;
constexpr size_t OFF_STAMP = 256 * 1024;
constexpr size_t OFF_ST2 = 768 * 1024;
constexpr size_t OFF_DATA = 1u << 20;
constexpr size_t WS_NEED  = OFF_DATA + 3ull * NR * SHARD;
constexpr unsigned long long DEADLINE_TICKS = 50000000ull;
constexpr int RW3 = 0x00020000;
constexpr int AUX_SYS = 17;
constexpr int AUX_DEV = 16;
constexpr int OOB = 0x7ffffff0;
constexpr unsigned X80 = 0x80808080u;

constexpr int LDS_ACC = MAXI * 1024 * 4;
constexpr int LDS_BYTES = LDS_ACC + 256;
#define OFF_TMO_ 61440
#define WS_NEED_ 0x3100000
#define RW3_ 0x20000
#define OFF_DATA_ 0x100000
static_assert(OFF_DATA_ == OFF_DATA && OFF_TMO_ == OFF_TMO && WS_NEED_ == WS_NEED && RW3_ == RW3, "literals");

#define CLOB "v64", "v65", "v66", "v67", "v68", "v69", "v70", "v71", "v72", "v73", "v74", "v75", "v76", "v77", "v78", "v79", "v80", "v81", "v82", "v83", "v84", "v85", "v86", "v87", "v88", "v89", "v90", "v91", "v92", "v93", "v94", "v95", "v96", "v97", "v98", "v99", "v100", "v101", "v102", "v103", "v104", "v105", "v106", "v107", "v108", "v109", "v110", "v111", "v112", "v113", "v114", "v115", "v116", "v117", "v118", "v119", "v120", "v121", "v122", "v123", "v124", "v125", "v126", "v127", "v128"
#define CLOB2 "v132", "v133", "v134", "v135", "v136", "v137", "v138", "v139", "v140", "v141", "v142", "v143", "v144", "v145", "v146", "v147", "v148", "v149", "v150", "v151", "v152", "v153", "v154", "v155", "v156", "v157", "v158", "v159", "v160", "v161", "v162", "v163"
struct Rec { char* ws; float* out; };
struct Recs { Rec r[8]; };

__device__ __forceinline__ unsigned umin4(v4u a) {
  return __builtin_elementwise_min(__builtin_elementwise_min(a.x, a.y), __builtin_elementwise_min(a.z, a.w));
}
__device__ __forceinline__ void tr4(unsigned a0, unsigned a1, unsigned a2, unsigned a3,
                                    unsigned& o0, unsigned& o1, unsigned& o2, unsigned& o3) {
  const unsigned t0 = __builtin_amdgcn_perm(a1, a0, 0x05010400u);
  const unsigned t1 = __builtin_amdgcn_perm(a1, a0, 0x07030602u);
  const unsigned t2 = __builtin_amdgcn_perm(a3, a2, 0x05010400u);
  const unsigned t3 = __builtin_amdgcn_perm(a3, a2, 0x07030602u);
  o0 = __builtin_amdgcn_perm(t2, t0, 0x05040100u);
  o1 = __builtin_amdgcn_perm(t2, t0, 0x07060302u);
  o2 = __builtin_amdgcn_perm(t3, t1, 0x05040100u);
  o3 = __builtin_amdgcn_perm(t3, t1, 0x07060302u);
}
__device__ __forceinline__ __amdgpu_buffer_rsrc_t rsrc(const void* p, int n) {
  return __builtin_amdgcn_make_buffer_rsrc((void*)p, 0, n, RW3);
}
constexpr int e_nld(int i) { return (i + 2 < 8 ? 16 : i + 2 < 8 + NXE ? 8 : 0) + (i == CLR ? 12 : 0); }
__host__ __device__ constexpr int h_last(int par) { return par + NPAR * ((NU - par + NPAR - 1) / NPAR - 1); }
static_assert(PIH < NU / NPAR && PI1H < NU / NPAR + 1 && 8 * CLR + WINW <= 56, "list ends with a stream item / sentinels land before the last w block leaves");
constexpr int e_wait(int i, int e) {
  const int W = i < 8 ? WINW : WINX;
  const int S = 8 * i + e - W;
  if (S < 0) return 63;
  int c = W - 1;
  for (int k = S / 8 + 1; k <= i; ++k) c += e_nld(k);
  return c > 63 ? 63 : c;
}
constexpr int e_tile(int ti) { return ti < NP ? NU + NQ + ti : NU + (ti - NP); }

__global__ void __launch_bounds__(512, 2)
ag_gemm_kernel(const int8_t* __restrict__ x, const int8_t* __restrict__ wm, char* myws, float* myout,
               const float* __restrict__ sxp, const float* __restrict__ swp,
               int rank, unsigned peers, unsigned xmap, unsigned pad, Recs R) {
  char* const pws = R.r[blockIdx.x & 7].ws;
  float* const pout = R.r[blockIdx.x & 7].out;
  asm volatile("" :: "s"(x), "s"(wm), "s"(myws), "s"(myout), "s"(sxp), "s"(swp), "s"(rank), "s"(peers), "s"(xmap), "s"(pws), "s"(pout));
  extern __shared__ __attribute__((aligned(16))) int lds[];
  const int tid = threadIdx.x;
  const int lane = tid & 63;
  const int wave = __builtin_amdgcn_readfirstlane(tid >> 6);
  const int xcd = (xmap >> (4 * (blockIdx.x & 7))) & 7;
  const int idx = blockIdx.x >> 3;
  unsigned* const cntp = (unsigned*)(myws + OFF_CNT + (size_t)blockIdx.x * 128) + wave;
  unsigned* const tmo = (unsigned*)(myws + OFF_TMO);
  const int lo = (lane & 15) * KD + (lane >> 4) * 16;

  if (LAY1(idx < 8 && xcd < 3)) {
    if (wave > (WARMP & 1)) return;
    const __amdgpu_buffer_rsrc_t rs = rsrc(x, wave ? 0 : SHARD);
    const __amdgpu_buffer_rsrc_t rd = rsrc(pws + OFF_DATA, wave ? 0 : 3 * NR * SHARD);
    const int g = idx;
    const int d0 = lane * 16, d1 = lane * 16 + 4096;
    unsigned call, dbase, oA, oB, t, u, to;
#define LD1(R, O, SO) "buffer_load_dwordx4 v[" #R ":" #R "+3], %[lo], %[rs], %[" SO "] offen offset:" #O "\n"
#define LDA(SO) LD1(64,0,SO) LD1(68,64,SO) LD1(72,128,SO) LD1(76,192,SO) LD1(80,256,SO) LD1(84,320,SO) LD1(88,384,SO) LD1(92,448,SO)
#define LDB(SO) LD1(96,0,SO) LD1(100,64,SO) LD1(104,128,SO) LD1(108,192,SO) LD1(112,256,SO) LD1(116,320,SO) LD1(120,384,SO) LD1(124,448,SO)
#define ST1(R, W, D, O, SO) "s_waitcnt vmcnt(" W ")\n" \
      "v_xor_b32 v[" #R "], %[x80], v[" #R "]\n" "v_xor_b32 v[" #R "+1], %[x80], v[" #R "+1]\n" \
      "v_xor_b32 v[" #R "+2], %[x80], v[" #R "+2]\n" "v_xor_b32 v[" #R "+3], %[x80], v[" #R "+3]\n" \
      "buffer_store_dwordx4 v[" #R ":" #R "+3], %[" #D "], %[rd], %[" SO "] offen offset:" #O " sc0 sc1\n" \
      "v_min3_u32 v129, v129, v[" #R "], v[" #R "+1]\n" "v_min3_u32 v129, v129, v[" #R "+2], v[" #R "+3]\n"
#define STA(W) ST1(64,W,d0,0,"oA") ST1(68,W,d0,1024,"oA") ST1(72,W,d0,2048,"oA") ST1(76,W,d0,3072,"oA") \
               ST1(80,W,d1,0,"oA") ST1(84,W,d1,1024,"oA") ST1(88,W,d1,2048,"oA") ST1(92,W,d1,3072,"oA")
#define STB(W) ST1(96,W,d0,0,"oB") ST1(100,W,d0,1024,"oB") ST1(104,W,d0,2048,"oB") ST1(108,W,d0,3072,"oB") \
               ST1(112,W,d1,0,"oB") ST1(116,W,d1,1024,"oB") ST1(120,W,d1,2048,"oB") ST1(124,W,d1,3072,"oB")
#define LDC(SO) LD1(132,0,SO) LD1(136,64,SO) LD1(140,128,SO) LD1(144,192,SO) LD1(148,256,SO) LD1(152,320,SO) LD1(156,384,SO) LD1(160,448,SO)
#define SX(R, I, SO) ST1(R+4*I, STR(WEND), dd##I, 1024*(I&3), SO)
#define dd0 d0
#define dd1 d0
#define dd2 d0
#define dd3 d0
#define dd4 d1
#define dd5 d1
#define dd6 d1
#define dd7 d1
#define ST1X(R, W, D, O, SO) ST1(R, W, D, O, SO)
#undef SX
#define SX(R, I, SO) ST1X(R+4*I, STR(WEND), dd##I, 1024*(I&3), SO)
#define STF(R, SO) SX(R,0,SO) SX(132,0,"u") SX(R,1,SO) SX(132,1,"u") SX(R,2,SO) SX(132,2,"u") SX(R,3,SO) SX(132,3,"u") \
      SX(R,4,SO) SX(132,4,"u") SX(R,5,SO) SX(132,5,"u") SX(R,6,SO) SX(132,6,"u") SX(R,7,SO) SX(132,7,"u")
#define NEXT(O, LBL) "s_add_u32 %[t], %[t], 0x10000\n" "s_cmp_ge_u32 %[t], %[lim]\n" "s_cbranch_scc1 " LBL "%=\n" \
      "s_add_u32 %[" O "], %[" O "], 0x20000\n"
#define STR2(x) #x
#define STR(x) STR2(x)
    asm volatile(
      "global_load_dword v128, %[cnt], off sc1\n"
      "s_lshl_b32 %[t], %[g], 9\n"
      "v_mov_b32 v129, -1\n"
      LDA("t")
      "s_add_u32 %[t], %[t], 0x10000\n"
      LDB("t")
      "s_add_u32 %[u], %[t], %[lim]\n" "s_sub_u32 %[u], %[u], 0x10000\n"
      LDC("u")
      "s_waitcnt vmcnt(24)\n"
      "v_readfirstlane_b32 %[call], v128\n"
      "s_add_u32 %[call], %[call], 1\n"
      "s_mul_hi_u32 %[u], %[call], 0xaaaaaaab\n"
      "s_lshr_b32 %[u], %[u], 1\n"
      "s_mul_i32 %[u], %[u], 3\n"
      "s_sub_u32 %[u], %[call], %[u]\n"
      "s_lshl_b32 %[u], %[u], 24\n"
      "s_add_u32 %[dbase], %[u], %[rk]\n"
      "s_lshl_b32 %[u], %[g], 13\n"
      "s_add_u32 %[oA], %[dbase], %[u]\n"
      "s_add_u32 %[oB], %[oA], 0x10000\n"
      "s_add_u32 %[u], %[oA], %[lim]\n"
      STA("23")
      NEXT("oA", "L_endB_") LDA("t")
      "L_loop_%=:\n"
      STB(STR(WLOOP) "-1") NEXT("oB", "L_endA_") LDB("t")
      STA(STR(WLOOP) "-1") NEXT("oA", "L_endB_") LDA("t")
      "s_branch L_loop_%=\n"
      "L_endA_%=:\n"
      STF(64, "oA")
      "s_branch L_done_%=\n"
      "L_endB_%=:\n"
      STF(96, "oB")
      "L_done_%=:\n"
      "v_cmp_eq_u32 vcc, 0, v129\n"
      "s_or_b32 %[to], vcc_lo, vcc_hi\n"
      : [call] "=&s"(call), [dbase] "=&s"(dbase), [oA] "=&s"(oA), [oB] "=&s"(oB), [t] "=&s"(t), [u] "=&s"(u), [to] "=&s"(to)
      : [lo] "v"(lo), [d0] "v"(d0), [d1] "v"(d1), [rs] "s"(rs), [rd] "s"(rd), [cnt] "v"(cntp),
        [g] "s"(g), [rk] "s"(rank * SHARD), [x80] "s"(X80), [lim] "s"((wave ? 4 + ((NU + NQ - 1) & 1) : NU + NQ - 1) * TILE)
      : "memory", "scc", "vcc", "v129", CLOB, CLOB2);
    if (wave) return;
    if (lane == 0) __hip_atomic_store(cntp, call, __ATOMIC_RELAXED, __HIP_MEMORY_SCOPE_AGENT);
    if (to)
    {
      asm volatile("s_waitcnt vmcnt(0)" ::: "memory");
      if (lane == 0)
        __hip_atomic_store((unsigned*)(pws + OFF_FLAG + (size_t)rank * 128) + g, call, __ATOMIC_RELAXED, __HIP_MEMORY_SCOPE_SYSTEM);
    }
    return;
  }

  if (LAY2(idx < 8 && xcd > 3)) {
    if (wave > (WARMP >> 1)) return;
    const int j = xcd - 4, g = idx;
    const unsigned prev = __hip_atomic_load(cntp, __ATOMIC_RELAXED, __HIP_MEMORY_SCOPE_AGENT);
    const __amdgpu_buffer_rsrc_t rw = rsrc(wm, wave ? 0 : KD * NS);
    const __amdgpu_buffer_rsrc_t rs = rsrc(x, wave ? 0 : SHARD);
    const int wlo = ((lane >> 3) * 16 + (g >> 1) * 128) * NS + j * 128 + (g & 1) * 64 + (lane & 7) * 8;
    v4u raw[2][8];
    auto load = [&](int i) __attribute__((always_inline)) {
      if (i < 8) {
#pragma unroll
        for (int t = 0; t < 16; ++t) {
          const v2u v = __builtin_amdgcn_raw_buffer_load_b64(rw, wlo + (t & 7) * NS, (i * 512 + (t & 8)) * NS, 0);
          raw[i & 1][t >> 1][(t & 1) * 2] = v.x;
          raw[i & 1][t >> 1][(t & 1) * 2 + 1] = v.y;
        }
      } else if (i < 8 + NXE) {
#pragma unroll
        for (int s = 0; s < 8; ++s)
          raw[i & 1][s] = __builtin_amdgcn_raw_buffer_load_b128(rs, lo + s * 64, e_tile(i - 8) * TILE + g * 512, 0);
      }
    };
    load(0);
    load(1);
    const unsigned call = __builtin_amdgcn_readfirstlane(prev) + 1u;
    const __amdgpu_buffer_rsrc_t rd = rsrc(pws + OFF_DATA + ((size_t)(call % 3u) * NR + (size_t)rank) * SHARD, wave ? 0 : SHARD);
#pragma unroll
    for (int i = 0; i < 8 + NXE; ++i) {
      v4u o[8];
      if (i < 8) {
#pragma unroll
        for (int q = 0; q < 4; ++q) {
          const v4u a = raw[i & 1][2 * q], b = raw[i & 1][2 * q + 1];
          unsigned c0, c1, c2, c3;
          tr4(a.x, a.z, b.x, b.z, c0, c1, c2, c3);
          o[0][q] = c0 ^ X80; o[1][q] = c1 ^ X80; o[2][q] = c2 ^ X80; o[3][q] = c3 ^ X80;
          tr4(a.y, a.w, b.y, b.w, c0, c1, c2, c3);
          o[4][q] = c0 ^ X80; o[5][q] = c1 ^ X80; o[6][q] = c2 ^ X80; o[7][q] = c3 ^ X80;
        }
      } else {
#pragma unroll
        for (int s = 0; s < 8; ++s) o[s] = raw[i & 1][s] ^ X80;
      }
      asm volatile("" ::: "memory");
      if (i == CLR) {
        const __amdgpu_buffer_rsrc_t rown = rsrc(myout, wave ? 0 : NR * MS * NS * 4);
        const int c_vo = ((lane >> 4) * NS + j * 128 + (g & 1) * 64 + 4 * (lane & 15)) * 4;
        const v4u ones = {~0u, ~0u, ~0u, ~0u};
#pragma unroll
        for (int st = 0; st < 3; ++st) {
          const int c_so = (int)(((peers >> (12 + 4 * ((j + 1 + st) & 3))) & 7u) * 32 + h_last(g >> 1)) << 15;
#pragma unroll
          for (int r = 0; r < 4; ++r) __builtin_amdgcn_raw_buffer_store_b128(ones, rown, c_vo, c_so + r * 4 * NS * 4, AUX_SYS);
        }
      }
      load(i + 2);
      const int so = (i < 8 ? i : e_tile(i - 8)) * TILE + g * SLICE;
#pragma unroll
      for (int e = 0; e < 8; ++e) {
        asm volatile("s_waitcnt vmcnt(%0)" :: "i"(e_wait(i, e)) : "memory");
        __builtin_amdgcn_raw_buffer_store_b128(o[e], rd, lane * 16 + e * 1024, so, AUX_SYS);
      }
    }
    asm volatile("s_waitcnt vmcnt(0)" ::: "memory");
    if (tid == 0) {
      __hip_atomic_store((unsigned*)(pws + OFF_FLAG + (size_t)rank * 128) + g, call, __ATOMIC_RELAXED, __HIP_MEMORY_SCOPE_SYSTEM);
      __hip_atomic_store(cntp, call, __ATOMIC_RELAXED, __HIP_MEMORY_SCOPE_AGENT);
    }
    return;
  }

  if (idx < 8) {
    const unsigned long long t0 = __builtin_amdgcn_s_memrealtime();
    const unsigned prev = __hip_atomic_load(cntp, __ATOMIC_RELAXED, __HIP_MEMORY_SCOPE_AGENT);
    const unsigned call = __builtin_amdgcn_readfirstlane(prev) + 1u;
    if (lane == 0) __hip_atomic_store(cntp, call, __ATOMIC_RELAXED, __HIP_MEMORY_SCOPE_AGENT);
    const unsigned zb = (call + 2u) % 3u;
    while (__builtin_amdgcn_s_memrealtime() - t0 < ZDELAY) __builtin_amdgcn_s_sleep(8);
    const __amdgpu_buffer_rsrc_t rz = rsrc(myws + OFF_DATA + (size_t)zb * NR * SHARD, NR * SHARD);
    const v4u z = {0u, 0u, 0u, 0u};
    for (int p = 0; p < 7; ++p) {
      const int base = (int)((peers >> (4 * p)) & 7u) * SHARD + idx * SLICE;
      const int n0 = p < 3 ? NU + NQ : 8;
      for (int t = 0; t < n0; ++t) __builtin_amdgcn_raw_buffer_store_b128(z, rz, tid * 16, base + t * TILE, AUX_DEV);
      if (p >= 3)
        for (int t = NU; t < NT; ++t) __builtin_amdgcn_raw_buffer_store_b128(z, rz, tid * 16, base + t * TILE, AUX_DEV);
    }
    return;
  }

  const unsigned long long t_end = __builtin_amdgcn_s_memrealtime() + DEADLINE_TICKS;
  const int sg = idx & 1;
  const int q = (idx - 8) >> 1;
  const int j = xcd & 3;
  const bool helper = xcd >= 4;
  int n, list;
  {
    const int st = q % 3, par = q / 3;
    const int nl = ((helper ? NU : NT) + q) / NWG;
    const int ns = ((helper ? NU : NU + NQ) - par + NPAR - 1) / NPAR;
    const int ni = ((helper ? 3 * NP : 4 * NXE) - q + NWG - 1) / NWG;
    const int p0 = ni < 1 ? 99 : helper ? PIH : PI0, p1 = ni < 2 ? 99 : helper ? PI1H : PI1;
    n = nl + ns + ni;
    const int m = lane - nl;
    const int si = m < p0 ? m : m - 1 < p1 ? m - 1 : m - 2;
    const int e = q + (m == p0 ? 0 : NWG);
    const int ti = helper ? e / 3 : e >> 2;
    const int pe = helper ? (j + 1 + e % 3) & 3 : e & 3;
    const int c_loc = 256 + rank * 32 + (NWG - 1 - q) + NWG * lane;
    const int c_str = (int)((peers >> (4 * st)) & 7u) * 32 + par + NPAR * si;
    const int c_int = (int)((peers >> (12 + 4 * pe)) & 7u) * 32 + (ti < NP ? NU + NQ + ti : NU + (ti - NP));
    list = m < 0 ? c_loc : (m == p0 || m - 1 == p1) ? c_int : c_str;
    if (lane == n - 1) list += 1024;
    if (lane >= n) list = 512;
  }
  const __amdgpu_buffer_rsrc_t rx = rsrc(x, SHARD);
  const int a_loc = lo + wave * 512;
  const int a_rem = lane * 16 + wave * SLICE;
  int* const lacc = lds;
  int* const ldone = lds + MAXI * 1024;
  {
    v4i z = {0, 0, 0, 0};
#pragma unroll
    for (int i = 0; i < (LDS_ACC / 16 + 511) / 512; ++i)
      if (i * 512 + tid < LDS_ACC / 16) ((v4i*)lacc)[i * 512 + tid] = z;
    if (tid < 64) ldone[tid] = 0;
  }
  const __amdgpu_buffer_rsrc_t rb = rsrc(myws, (int)WS_NEED);
  const __amdgpu_buffer_rsrc_t rw = rsrc(wm, KD * NS);
  const unsigned owner = helper ? __builtin_amdgcn_readfirstlane((peers >> (12 + 4 * j)) & 7u) : (unsigned)rank;
  unsigned call, dbase;
  int dead = 0;
  char* rfp = (helper ? pws : myws) + OFF_RFLAG + (size_t)(peers >> 28) * RFS;
  asm volatile("" : "+s"(rfp));
  const int b_so = helper ? (int)owner * SHARD + wave * TILE + sg * SLICE : wave * 512 * NS;
  const int b_vo = helper ? (lane & 1) * 4096 + (lane >> 4) * 128 + ((lane & 15) >> 1) * 16
                          : ((lane >> 4) * 16) * NS + (2 * j + sg) * 64 + (lane & 15) * 4;
  const int b_g = lane < NW ? (int)OFF_FLAG + (int)owner * 128 + lane * 4
                : lane < NW + 4 ? (lane - NW) * 2 * SLICE + (SLICE - 4) : OOB;

  const int e_go = ((4 * (lane >> 4) + (wave & 3)) * NS + (helper ? (int)(peers >> 28) * 128 : j * 128) + sg * 64 + 4 * (lane & 15)) * 4;
  const __amdgpu_buffer_rsrc_t ro = rsrc(pout, NR * MS * NS * 4);
  int k = 0, e = ALT ? wave >> 2 : wave < 4 ? 0 : 99;
#define TL1 ""
#define TL2 ""
#define TS(R) ""
  {
    const int g1 = lane == 0 ? wave * SLICE + GOFF : OOB;
    const int fl = lane == 0 ? (int)OFF_FLAG + wave * 4 : OOB;
    const int l4 = lane * 4;
    const int chk = helper || wave < 4 ? 0 : (int)(((peers >> (12 + 4 * ((j + 1 + q % 3) & 3))) & 7u) * 32 + h_last(q / 3)) << 15;
    const int rfo = (int)OFF_RFLAG + j * RFS + ((idx - 8) * 4 + (wave & 3)) * FSTR;
    const int le = ((wave & 3) * 64 + lane) * 4 + (wave >> 2) * 0x1000 * ALT;
    const int led = LDS_ACC + (wave >> 2) * 4 * ALT;
#define vSC  "v23"
#define vT   "v32"
#define vP   "v33"
#define vG0  "v34"
#define vG1  "v35"
#define vLA  "v36"
#define vLD  "v37"
#define vEA  "v38"
#define vED  "v39"
#define vONE "v40"
#define vBAD "v41"
#define vDN  "v42"
#define vP60 "v43"
#define sC   "s66"
#define sC1  "s67"
#define sC2  "s68"
#define sFIN "s69"
#define sMISS "s70"
#define sSP  "s71"
#define sT   "s94"
#define sU   "s95"
#define sRDY "s98"
#define sNFN "s99"
#define GATEOFF(C) "s_and_b32 " sT ", " C ", 255\n" "s_lshl_b32 " sT ", " sT ", 16\n" "s_add_u32 " sT ", " sT ", %[dbase]\n" \
      "s_bitcmp1_b32 " C ", 10\n" "s_cselect_b32 " sU ", %[ge], 0\n" "s_sub_u32 " sT ", " sT ", " sU "\n"
#define LDR(Y, S, O, SO) "buffer_load_dwordx4 v[" #Y "+4*" #S ":" #Y "+4*" #S "+3], %[arem], s[76:79], " SO " offen offset:" #O " sc0 sc1\n"
#define LDL(Y, S, O) "buffer_load_dwordx4 v[" #Y "+4*" #S ":" #Y "+4*" #S "+3], %[aloc], %[rx], " sT " offen offset:" #O "\n"
#define ISSUE(Y, GY, NF, XM, L) \
      GATEOFF(sC2) \
      "s_bitcmp1_b32 " sC1 ", 9\n" "s_cselect_b32 s78, 0, " sRDY "\n" \
      "s_and_b32 " sU ", " sC2 ", 0x300\n" "s_cselect_b32 s74, 0, s78\n" \
      "buffer_load_dword " GY ", %[g1], s[72:75], " sT " offen sc0 sc1\n" \
      "s_bitcmp1_b32 " sC1 ", 8\n" "s_cbranch_scc1 Lloc" L #Y "_%=\n" \
      "s_and_b32 " sT ", " sC1 ", 255\n" "s_lshl_b32 " sT ", " sT ", 16\n" "s_add_u32 " sT ", " sT ", %[dbase]\n" \
      "s_add_u32 " sU ", " sT ", 0x1000\n" \
      LDR(Y,0,0,sT) LDR(Y,1,1024,sT) LDR(Y,2,2048,sT) LDR(Y,3,3072,sT) LDR(Y,4,0,sU) LDR(Y,5,1024,sU) LDR(Y,6,2048,sU) LDR(Y,7,3072,sU) \
      "s_mov_b32 " XM ", 0x80808080\n" \
      "s_bfe_u32 " sT ", " sC1 ", 0x30005\n" "s_bitcmp1_b32 " sFIN ", " sT "\n" "s_cselect_b64 " NF ", 0, -1\n" \
      "s_branch Liss" L #Y "_%=\n" \
      "Lloc" L #Y "_%=:\n" \
      "s_and_b32 " sT ", " sC1 ", 31\n" "s_lshl_b32 " sT ", " sT ", 16\n" \
      LDL(Y,0,0) LDL(Y,1,64) LDL(Y,2,128) LDL(Y,3,192) LDL(Y,4,256) LDL(Y,5,320) LDL(Y,6,384) LDL(Y,7,448) \
      "s_mov_b32 " XM ", 0\n" "s_mov_b64 " NF ", 0\n" \
      "Liss" L #Y "_%=:\n"
#define MF(X, S, D) "v_mfma_i32_16x16x64_i8 v[48+4*" #D ":51+4*" #D "], v[" #X "+4*" #S ":" #X "+4*" #S "+3], v[128+16*" #S "+4*" #D ":131+16*" #S "+4*" #D "], v[48+4*" #D ":51+4*" #D "]\n"
#define CHUNK(X, S, NF, XM) \
      "s_bitcmp1_b32 " sMISS ", " #S "\n" "s_cbranch_scc0 Lc" #X "_" #S "_%=\n" \
      "s_waitcnt vmcnt(17-" #S ")\n" \
      "v_min_u32 " vT ", v[" #X "+4*" #S "], v[" #X "+4*" #S "+1]\n" \
      "v_min3_u32 " vT ", " vT ", v[" #X "+4*" #S "+2], v[" #X "+4*" #S "+3]\n" \
      "v_cmp_eq_u32 vcc, 0, " vT "\n" \
      "v_xor_b32 v[" #X "+4*" #S "], " XM ", v[" #X "+4*" #S "]\n" "v_xor_b32 v[" #X "+4*" #S "+1], " XM ", v[" #X "+4*" #S "+1]\n" \
      "v_xor_b32 v[" #X "+4*" #S "+2], " XM ", v[" #X "+4*" #S "+2]\n" "v_xor_b32 v[" #X "+4*" #S "+3], " XM ", v[" #X "+4*" #S "+3]\n" \
      "s_and_b64 vcc, vcc, " NF "\n" "s_cbranch_scc1 Lc" #X "_" #S "_%=\n" \
      "s_bitset0_b32 " sMISS ", " #S "\n" \
      MF(X,S,0) MF(X,S,1) MF(X,S,2) MF(X,S,3) \
      "Lc" #X "_" #S "_%=:\n"
#define RLD(X, S, O, SO) "s_bitcmp1_b32 " sMISS ", " #S "\n" "s_cbranch_scc0 Lr" #X "_" #S "_%=\n" \
      "buffer_load_dwordx4 v[" #X "+4*" #S ":" #X "+4*" #S "+3], %[arem], %[rb], " SO " offen offset:" #O " sc0 sc1\n" "Lr" #X "_" #S "_%=:\n"
#define DSA(I) "ds_add_u32 " vLA ", v[48+" #I "] offset:256*" #I "\n"
#define SILU(R) "v_cvt_f32_i32 v[" #R "], v[" #R "]\n" "v_mul_f32 v[" #R "], " vSC ", v[" #R "]\n" \
      "v_med3_f32 v[" #R "+4], v[" #R "], %[m60], " vP60 "\n" "v_mul_f32 v[" #R "+4], 0xbfb8aa3b, v[" #R "+4]\n"
#define DUTY(L, NO, PRE) \
      "s_cmp_ge_i32 %[e], %[n]\n" "s_cbranch_scc1 " NO "\n" \
      "ds_read_b32 " vDN ", " vED "\n" \
      "ds_read_b32 v24, " vEA "\n" "ds_read_b32 v25, " vEA " offset:1024\n" \
      "ds_read_b32 v26, " vEA " offset:2048\n" "ds_read_b32 v27, " vEA " offset:3072\n" \
      "ds_read_b32 " vT ", " vBAD "\n" \
      "v_readlane_b32 " sU ", %[list], %[e]\n" \
      "s_waitcnt lgkmcnt(0)\n" \
      "v_readfirstlane_b32 " sT ", " vDN "\n" "s_cmp_lg_u32 " sT ", 8\n" "s_cbranch_scc1 " NO "\n" \
      SILU(24) SILU(25) SILU(26) SILU(27) \
      "v_exp_f32 v28, v28\n" "v_exp_f32 v29, v29\n" "v_exp_f32 v30, v30\n" "v_exp_f32 v31, v31\n" \
      "v_cmp_ne_u32 vcc, 0, " vT "\n" \
      "v_add_f32 v28, 1.0, v28\n" "v_add_f32 v29, 1.0, v29\n" "v_add_f32 v30, 1.0, v30\n" "v_add_f32 v31, 1.0, v31\n" \
      "v_rcp_f32 v28, v28\n" "v_rcp_f32 v29, v29\n" "v_rcp_f32 v30, v30\n" "v_rcp_f32 v31, v31\n" \
      "s_and_b32 " sU ", " sU ", 255\n" "s_lshl_b32 " sU ", " sU ", 15\n" \
      "v_mul_f32 v44, v24, v28\n" "v_mul_f32 v45, v25, v29\n" "v_mul_f32 v46, v26, v30\n" "v_mul_f32 v47, v27, v31\n" \
      "s_cbranch_vccz Lok" L "_%=\n" \
      "v_mov_b32 v44, 0x7fc00000\n" "v_mov_b32 v45, 0x7fc00000\n" "v_mov_b32 v46, 0x7fc00000\n" "v_mov_b32 v47, 0x7fc00000\n" \
      "Lok" L "_%=:\n" \
      PRE \
      "buffer_store_dwordx4 v[44:47], %[ego], %[ro], " sU " offen sc0 sc1\n" \
      TL2 \
      "s_add_u32 %[e], %[e], 1+" STR(ALT) "\n" \
      "v_add_u32 " vEA ", 0x1000*(1+" STR(ALT) "), " vEA "\n" "v_add_u32 " vED ", 4*(1+" STR(ALT) "), " vED "\n"
#define EXPIRE(L) \
      "s_add_u32 " sSP ", " sSP ", 1\n" "s_and_b32 " sT ", " sSP ", 255\n" "s_cbranch_scc1 Lx" L "_%=\n" \
      "s_memrealtime s[96:97]\n" "s_mov_b32 " sT ", " STR(OFF_TMO_) "\n" \
      "buffer_load_dword " vT ", off, %[rb], " sT " sc1\n" \
      "s_waitcnt vmcnt(0) lgkmcnt(0)\n" \
      "v_readfirstlane_b32 " sT ", " vT "\n" "s_cmp_eq_u32 " sT ", %[call]\n" "s_cbranch_scc1 Ldead_%=\n" \
      "s_sub_u32 " sT ", %[tlo], s96\n" "s_subb_u32 " sT ", %[thi], s97\n" "s_cbranch_scc1 Ldead_%=\n" \
      "Lx" L "_%=:\n"
#define WNX64 "Lrt96_%="
#define WNX96 "Ldrain_%="
#define WRT(N) "s_cmp_lg_u32 s56, 0\n" "s_cbranch_scc1 " N "\n"
#define WFIRST "Lrt64_%="
#define STEP(X, GX, NFX, XMX, Y, GY, NFY, XMY) \
      "Lstep" #X "_%=:\n" \
      "s_add_u32 " sT ", %[k], 2\n" \
      "s_mov_b32 " sC ", " sC1 "\n" "s_mov_b32 " sC1 ", " sC2 "\n" \
      "v_readlane_b32 " sC2 ", %[list], " sT "\n" \
      "s_bfe_u32 " sT ", " sC1 ", 0x30005\n" "s_lshr_b32 " sT ", " sFIN ", " sT "\n" "s_and_b32 " sT ", " sT ", 1\n" \
      "s_and_b32 " sU ", " sC1 ", 0x300\n" "s_or_b32 " sT ", " sT ", " sU "\n" \
      "s_waitcnt vmcnt(9)\n" \
      "v_readfirstlane_b32 " sU ", " GX "\n" "s_or_b32 " sT ", " sT ", " sU "\n" \
      "s_cselect_b32 " sRDY ", " STR(WS_NEED_) ", 0\n" \
      ISSUE(Y, GY, NFY, XMY, "a") \
      "v_mov_b64 v[48:49], 0\n" "v_mov_b64 v[50:51], 0\n" "v_mov_b64 v[52:53], 0\n" "v_mov_b64 v[54:55], 0\n" \
      "v_mov_b64 v[56:57], 0\n" "v_mov_b64 v[58:59], 0\n" "v_mov_b64 v[60:61], 0\n" "v_mov_b64 v[62:63], 0\n" \
      "s_mov_b32 " sMISS ", 0xff\n" "s_mov_b32 " sNFN ", -1\n" \
      "Latt" #X "_%=:\n" \
      CHUNK(X,0,NFX,XMX) CHUNK(X,1,NFX,XMX) CHUNK(X,2,NFX,XMX) CHUNK(X,3,NFX,XMX) \
      CHUNK(X,4,NFX,XMX) CHUNK(X,5,NFX,XMX) CHUNK(X,6,NFX,XMX) CHUNK(X,7,NFX,XMX) \
      "s_cmp_eq_u32 " sMISS ", 0\n" "s_cbranch_scc1 Lfull" #X "_%=\n" \
      "Lrt" #X "_%=:\n" \
        \
      "s_bfe_u32 " sT ", " sC ", 0x30005\n" "s_lshl_b32 " sT ", " sT ", 7\n" \
      "buffer_load_dword " vP ", %[fl], %[rb], " sT " offen sc0 sc1\n" \
      "s_and_b32 " sT ", " sC ", 255\n" "s_lshl_b32 " sT ", " sT ", 16\n" "s_add_u32 " sT ", " sT ", %[dbase]\n" \
      "s_add_u32 " sU ", " sT ", 0x1000\n" \
      RLD(X,0,0,sT) RLD(X,1,1024,sT) RLD(X,2,2048,sT) RLD(X,3,3072,sT) RLD(X,4,0,sU) RLD(X,5,1024,sU) RLD(X,6,2048,sU) RLD(X,7,3072,sU) \
      "s_cmp_eq_u32 " sNFN ", 0\n" "s_cselect_b64 " NFX ", 0, -1\n" \
      EXPIRE("r" #X) \
      WRT(WNX##X) \
      "s_waitcnt vmcnt(0)\n" \
      "v_cmp_le_u32 vcc, %[call], " vP "\n" \
      "s_bitcmp1_b32 vcc_lo, 0\n" "s_cbranch_scc0 Latt" #X "_%=\n" \
      "s_mov_b32 " sNFN ", 0\n" "s_bfe_u32 " sT ", " sC ", 0x30005\n" "s_bitset1_b32 " sFIN ", " sT "\n" \
      "s_branch Latt" #X "_%=\n" \
      "Lfull" #X "_%=:\n" \
      TS("s[58:59]") \
      "s_nop 6\n" \
      DSA(0) DSA(1) DSA(2) DSA(3) DSA(4) DSA(5) DSA(6) DSA(7) DSA(8) DSA(9) DSA(10) DSA(11) DSA(12) DSA(13) DSA(14) DSA(15) \
      "s_mov_b64 exec, 1\n" "ds_add_u32 " vLD ", " vONE "\n" "s_mov_b64 exec, -1\n" \
      TL1 \
      "s_add_u32 %[k], %[k], 1\n" \
      "v_add_u32 " vLA ", 0x1000, " vLA "\n" "v_add_u32 " vLD ", 4, " vLD "\n" \
      "s_cmp_ge_i32 %[k], %[n]\n" "s_cbranch_scc1 Ldrain_%=\n" \
      "s_cmp_lg_u32 " sRDY ", 0\n" "s_cbranch_scc1 Ltail" #X "_%=\n" \
      TFIRST(#X) \
      "Lpoll" #X "_%=:\n" \
      GATEOFF(sC1) \
      "buffer_load_dword " vP ", %[g1], %[rb], " sT " offen sc0 sc1\n" \
      DUTY("p" #X, "Lpn" #X "_%=", "") \
      "s_waitcnt vmcnt(1)\n" "s_branch Lpw" #X "_%=\n" \
      "Lpn" #X "_%=:\n" "s_waitcnt vmcnt(0)\n" \
      "Lpw" #X "_%=:\n" \
      "v_readfirstlane_b32 " sT ", " vP "\n" "s_cmp_lg_u32 " sT ", 0\n" "s_cbranch_scc1 Lgo" #X "_%=\n" \
      "s_and_b32 " sT ", " sSP ", 15\n" "s_cbranch_scc1 Lnf" #X "_%=\n" \
      "s_bfe_u32 " sT ", " sC1 ", 0x30005\n" "s_lshl_b32 " sT ", " sT ", 7\n" \
      "buffer_load_dword " vP ", %[fl], %[rb], " sT " offen sc0 sc1\n" \
      "s_waitcnt vmcnt(0)\n" \
      "v_cmp_le_u32 vcc, %[call], " vP "\n" \
      "s_bitcmp1_b32 vcc_lo, 0\n" "s_cbranch_scc0 Lnf" #X "_%=\n" \
      "s_bfe_u32 " sT ", " sC1 ", 0x30005\n" "s_bitset1_b32 " sFIN ", " sT "\n" \
      "s_branch Lgo" #X "_%=\n" \
      "Lnf" #X "_%=:\n" \
      EXPIRE("p" #X) \
      "s_sleep " STR(SLP) "\n" \
      "s_branch Lpoll" #X "_%=\n" \
      "Lgo" #X "_%=:\n" \
      TS("s[56:57]") \
      "s_mov_b32 " sRDY ", " STR(WS_NEED_) "\n" \
      ISSUE(Y, GY, NFY, XMY, "b") \
      "Ltail" #X "_%=:\n" \
      DUTY("t" #X, "Ltn" #X "_%=", "") \
      "s_branch Lstep" #Y "_%=\n" \
      "Ltn" #X "_%=:\n" \
      "buffer_store_dwordx4 v[44:47], %[ego], s[84:87], 0 offen\n" \
      "s_branch Lstep" #Y "_%=\n"
#define TMISS ""
#define TFIRST(X) ""
#define BL1(S, T) "buffer_load_dword v[128+16*" #S "+4*(" #T "&3)+(" #T ">>2)], %[bvo], %[rw], " sT " offen offset:512*(" #T "&7)\n"
#define BLD(S) BL1(S,0) BL1(S,1) BL1(S,2) BL1(S,3) BL1(S,4) BL1(S,5) BL1(S,6) BL1(S,7) "s_add_u32 " sT ", " sT ", 0x1000\n" \
      BL1(S,8) BL1(S,9) BL1(S,10) BL1(S,11) BL1(S,12) BL1(S,13) BL1(S,14) BL1(S,15) "s_add_u32 " sT ", " sT ", 0x7000\n"
#define BT1(S, D) \
      "v_perm_b32 v24, v[132+16*" #S "+" #D "], v[128+16*" #S "+" #D "], s80\n" "v_perm_b32 v25, v[132+16*" #S "+" #D "], v[128+16*" #S "+" #D "], s81\n" \
      "v_perm_b32 v26, v[140+16*" #S "+" #D "], v[136+16*" #S "+" #D "], s80\n" "v_perm_b32 v27, v[140+16*" #S "+" #D "], v[136+16*" #S "+" #D "], s81\n" \
      "v_perm_b32 v[128+16*" #S "+" #D "], v26, v24, s82\n" "v_perm_b32 v[132+16*" #S "+" #D "], v26, v24, s83\n" \
      "v_perm_b32 v[136+16*" #S "+" #D "], v27, v25, s82\n" "v_perm_b32 v[140+16*" #S "+" #D "], v27, v25, s83\n"
#define BTR(S, W) "s_waitcnt vmcnt(" #W ")\n" BT1(S,0) BT1(S,1) BT1(S,2) BT1(S,3)
#define HL1(S, C) "buffer_load_dwordx4 v[128+16*" #S "+4*" #C ":131+16*" #S "+4*" #C "], %[bvo], %[rb], " sT " offen offset:1024*" #C "+512*(" #S "&1) sc0 sc1\n"
#define HLD(S, S1) HL1(S,0) HL1(S,1) HL1(S,2) HL1(S,3) HL1(S1,0) HL1(S1,1) HL1(S1,2) HL1(S1,3) "s_add_u32 " sT ", " sT ", 0x4000\n"
#define HC1(R) "v_min3_u32 " vT ", " vT ", v[" #R "], v[" #R "+1]\n" "v_min3_u32 " vT ", " vT ", v[" #R "+2], v[" #R "+3]\n" \
      "v_xor_b32 v[" #R "], s80, v[" #R "]\n" "v_xor_b32 v[" #R "+1], s80, v[" #R "+1]\n" \
      "v_xor_b32 v[" #R "+2], s80, v[" #R "+2]\n" "v_xor_b32 v[" #R "+3], s80, v[" #R "+3]\n"
#define HCK(S, W) "s_waitcnt vmcnt(" #W ")\n" HC1(128+16*S) HC1(132+16*S) HC1(136+16*S) HC1(140+16*S)
#define CALL(W) "s_waitcnt vmcnt(" #W ")\n" \
      "v_readfirstlane_b32 %[call], " vP "\n" "s_add_u32 %[call], %[call], 1\n" \
      "s_mul_hi_u32 " sU ", %[call], 0xaaaaaaab\n" "s_lshr_b32 " sU ", " sU ", 1\n" "s_mul_i32 " sU ", " sU ", 3\n" \
      "s_sub_u32 " sU ", %[call], " sU "\n" "s_lshl_b32 " sU ", " sU ", 24\n" "s_add_u32 %[dbase], " sU ", " STR(OFF_DATA_) "\n"
#define WARM "v_mov_b32 " vDN ", 8\n" "s_mov_b32 s56, 1\n" "s_mov_b32 s57, %[e]\n" "s_sub_u32 %[e], %[n], 1\n" \
      "s_mov_b64 exec, 0\n" "s_mov_b32 " sMISS ", 0xff\n" "s_branch " WFIRST "\n"
#define WEND_ "s_cmp_lg_u32 s56, 0\n" "s_cbranch_scc1 Lback_%=\n" \
      "s_cmp_lg_u32 %[dead], 0\n" "s_cbranch_scc1 Lslow_%=\n" "s_endpgm\n" \
      "Lback_%=:\n" "s_mov_b32 s56, 0\n" "s_mov_b64 s[88:89], 0\n" "s_mov_b32 %[e], s57\n" \
      "s_mov_b64 exec, 1\n" "v_mov_b32 " vT ", %[call]\n" "global_store_dword %[cnt], " vT ", off sc1\n" "s_mov_b64 exec, -1\n" \
      "s_cmp_ge_u32 %[hlp], 4\n" "s_cbranch_scc1 Lhp_%=\n" "s_branch Lrun_%=\n" \
      "Lslow_%=:\n"
    asm volatile(
      "s_nop 4\n"
      "s_sleep " STR(CDELAY) "\n"
      "global_load_dword " vP ", %[cnt], off sc1\n"
      "s_load_dword s96, %[sx], 0\n" "s_load_dword s97, %[sw], 0\n"
      "v_readlane_b32 " sC1 ", %[list], 0\n" "v_readlane_b32 " sC2 ", %[list], 1\n"
      "s_and_b32 " sT ", " sC1 ", 31\n" "s_lshl_b32 " sT ", " sT ", 16\n"
      LDL(64,0,0) LDL(64,1,64) LDL(64,2,128) LDL(64,3,192) LDL(64,4,256) LDL(64,5,320) LDL(64,6,384) LDL(64,7,448)
      "s_mov_b64 s[72:73], %[wsp]\n" "s_mov_b32 s75, " STR(RW3_) "\n"
      "s_mov_b64 s[76:77], %[wsp]\n" "s_mov_b32 s79, " STR(RW3_) "\n"
      "s_mov_b64 s[84:85], %[wsp]\n" "s_mov_b32 s86, 0\n" "s_mov_b32 s87, " STR(RW3_) "\n"
      "s_mov_b64 s[88:89], 0\n" "s_mov_b32 s92, 0\n"
      "s_mov_b32 " sFIN ", 0\n" "s_mov_b32 " sSP ", 1\n"
      "v_mov_b32 " vG0 ", 0\n" "v_mov_b32 " vLA ", %[l4]\n" "v_mov_b32 " vLD ", %[lacc]\n"
      "v_mov_b32 " vEA ", %[le]\n" "v_mov_b32 " vED ", %[led]\n"
      "v_mov_b32 " vONE ", 1\n" "v_mov_b32 " vBAD ", %[lbad]\n" "v_mov_b32 " vP60 ", 0x42700000\n"
      "s_cmp_ge_u32 %[hlp], 4\n" "s_cbranch_scc1 Lhelp_%=\n"
      "s_mov_b32 " sT ", %[bso]\n"
      BLD(0) BLD(1) BLD(2)
      "s_mov_b32 s80, 0x05010400\n" "s_mov_b32 s81, 0x07030602\n" "s_mov_b32 s82, 0x05040100\n" "s_mov_b32 s83, 0x07060302\n"
      "s_waitcnt lgkmcnt(0)\n" "s_barrier\n"
      "v_mov_b32 " vSC ", s96\n" "v_mul_f32 " vSC ", s97, " vSC "\n"
      CALL(56)
      BTR(0,32) BLD(3) BTR(1,32) BLD(4) BTR(2,32) BLD(5) BTR(3,32) BLD(6) BTR(4,32) BLD(7) BTR(5,32) BTR(6,16) BTR(7,0)
      WARM
      "s_branch Lrun_%=\n"
      "Lhelp_%=:\n"
      "s_mov_b32 s80, 0x80808080\n"
      "s_waitcnt lgkmcnt(0)\n" "s_barrier\n"
      "v_mov_b32 " vSC ", s96\n" "v_mul_f32 " vSC ", s97, " vSC "\n"
      "v_mov_b32 v24, %[bg]\n" "v_mov_b32 v25, 1\n"
      CALL(8)
      "s_add_u32 s81, %[dbase], %[bso]\n"
      "s_mov_b64 exec, 0xff\n" "v_mov_b32 v25, %[call]\n"
      "s_mov_b64 exec, 0xf00\n" "v_add_u32 v24, s81, v24\n"
      "s_mov_b64 exec, -1\n"
      WARM
      "Lhp_%=:\n"
      "buffer_load_dword " vP ", v24, %[rb], 0 offen sc0 sc1\n"
      "s_waitcnt vmcnt(0)\n"
      "v_cmp_le_u32 vcc, v25, " vP "\n"
      "s_and_b32 " sT ", vcc_lo, 0xff\n" "s_cmp_eq_u32 " sT ", 0xff\n" "s_cselect_b64 s[82:83], 0, -1\n" "s_cbranch_scc1 Lhl_%=\n"
      "s_and_b32 " sT ", vcc_lo, 0xf00\n" "s_cmp_eq_u32 " sT ", 0xf00\n" "s_cbranch_scc0 Lhn_%=\n"
      "Lhl_%=:\n"
      "s_mov_b32 " sT ", s81\n" "v_mov_b32 " vT ", -1\n"
      HLD(0,1) HLD(2,3) HLD(4,5) HLD(6,7)
      HCK(0,28) HCK(1,24) HCK(2,20) HCK(3,16) HCK(4,12) HCK(5,8) HCK(6,4) HCK(7,0)
      "v_cmp_eq_u32 vcc, 0, " vT "\n"
      "s_and_b64 vcc, vcc, s[82:83]\n" "s_cbranch_scc0 Lrun_%=\n"
      "Lhn_%=:\n"
      EXPIRE("h")
      "s_sleep " STR(SLP) "\n"
      "s_branch Lhp_%=\n"
      "Lrun_%=:\n"
      TS("s[64:65]")
      STEP(64, vG0, "s[88:89]", "s92", 96, vG1, "s[90:91]", "s93")
      STEP(96, vG1, "s[90:91]", "s93", 64, vG0, "s[88:89]", "s92")
      "Ldead_%=:\n"
      "s_mov_b32 %[dead], 1\n"
      "s_branch Lend_%=\n"
      "Ldrain_%=:\n"
      DUTY("d", "Ldn_%=", "s_waitcnt vmcnt(0)\n")
      "Ldn_%=:\n"
      "s_cmp_lt_i32 %[e], %[n]\n" "s_cbranch_scc1 Ldrain_%=\n"
      "s_cmp_eq_u32 %[chk], 0\n" "s_cbranch_scc1 Lamb_%=\n"
      "Lck_%=:\n"
      "buffer_load_dwordx4 v[44:47], %[ego], %[ro], %[chk] offen sc0 sc1\n"
      "s_waitcnt vmcnt(0)\n"
      "Lamb_%=:\n"
      "v_max3_u32 " vT ", v44, v45, v46\n" "v_max_u32 " vT ", " vT ", v47\n"
      "v_cmp_eq_u32 vcc, -1, " vT "\n"
      "s_cbranch_vccz Lend_%=\n"
      "s_cmp_eq_u32 %[chk], 0\n" "s_cselect_b32 %[dead], 2, 0\n" "s_cbranch_scc1 Lend_%=\n"
      "s_and_b32 " sT ", " sSP ", 15\n" "s_cbranch_scc1 Lcn_%=\n"
      "buffer_load_dword " vP ", off, %[rb], %[rfo] sc0 sc1\n"
      "s_waitcnt vmcnt(0)\n"
      "v_readfirstlane_b32 " sT ", " vP "\n" "s_cmp_ge_u32 " sT ", %[call]\n" "s_cbranch_scc1 Lend_%=\n"
      "Lcn_%=:\n"
      EXPIRE("c")
      "s_sleep " STR(SLP) "\n"
      "s_branch Lck_%=\n"
      "Lend_%=:\n"
      WEND_
      : [k] "+s"(k), [e] "+s"(e), [dead] "+s"(dead),
        [call] "=&s"(call), [dbase] "=&s"(dbase)
      : [list] "v"(list), [aloc] "v"(a_loc), [arem] "v"(a_rem), [g1] "v"(g1), [fl] "v"(fl), [l4] "v"(l4), [le] "v"(le), [ego] "v"(e_go),
        [cnt] "v"(cntp), [bvo] "v"(b_vo), [bg] "v"(b_g), [bso] "s"(b_so), [hlp] "s"(xcd), [sx] "s"(sxp), [sw] "s"(swp),
        [wsp] "s"(myws), [rb] "s"(rb), [rx] "s"(rx), [rw] "s"(rw), [ro] "s"(ro), [n] "s"(n),
        [chk] "s"(chk), [rfo] "s"(rfo), [m60] "s"(-60.0f), [ge] "s"(helper ? GEARLYH : GEARLY), [lacc] "i"(LDS_ACC), [led] "s"(led), [lbad] "i"(LDS_ACC + 252), [tlo] "s"((unsigned)t_end), [thi] "s"((unsigned)(t_end >> 32))
      : "memory", "scc", "vcc",
        "v23", "v24", "v25", "v26", "v27", "v28", "v29", "v30", "v31", "v32", "v33", "v34", "v35", "v36", "v37", "v38", "v39", "v40",
        "v41", "v42", "v43", "v44", "v45", "v46", "v47", "v48", "v49", "v50", "v51", "v52", "v53", "v54", "v55", "v56", "v57", "v58",
        "v59", "v60", "v61", "v62", "v63", "v64", "v65", "v66", "v67", "v68", "v69", "v70", "v71", "v72", "v73", "v74", "v75", "v76",
        "v77", "v78", "v79", "v80", "v81", "v82", "v83", "v84", "v85", "v86", "v87", "v88", "v89", "v90", "v91", "v92", "v93", "v94",
        "v95", "v96", "v97", "v98", "v99", "v100", "v101", "v102", "v103", "v104", "v105", "v106", "v107", "v108", "v109", "v110", "v111",
        "v112", "v113", "v114", "v115", "v116", "v117", "v118", "v119", "v120", "v121", "v122", "v123", "v124", "v125", "v126", "v127",
        "v128", "v129", "v130", "v131", "v132", "v133", "v134", "v135", "v136", "v137", "v138", "v139", "v140", "v141", "v142", "v143",
        "v144", "v145", "v146", "v147", "v148", "v149", "v150", "v151", "v152", "v153", "v154", "v155", "v156", "v157", "v158", "v159",
        "v160", "v161", "v162", "v163", "v164", "v165", "v166", "v167", "v168", "v169", "v170", "v171", "v172", "v173", "v174", "v175",
        "v176", "v177", "v178", "v179", "v180", "v181", "v182", "v183", "v184", "v185", "v186", "v187", "v188", "v189", "v190", "v191",
        "v192", "v193", "v194", "v195", "v196", "v197", "v198", "v199", "v200", "v201", "v202", "v203", "v204", "v205", "v206", "v207",
        "v208", "v209", "v210", "v211", "v212", "v213", "v214", "v215", "v216", "v217", "v218", "v219", "v220", "v221", "v222", "v223",
        "v224", "v225", "v226", "v227", "v228", "v229", "v230", "v231", "v232", "v233", "v234", "v235", "v236", "v237", "v238", "v239",
        "v240", "v241", "v242", "v243", "v244", "v245", "v246", "v247", "v248", "v249", "v250", "v251", "v252", "v253", "v254", "v255",
        "s56", "s57", "s58", "s59", "s60", "s61", "s62", "s63", "s64", "s65", "s66", "s67", "s68", "s69", "s70", "s71", "s72", "s73",
        "s74", "s75", "s76", "s77", "s78", "s79", "s80", "s81", "s82", "s83", "s84", "s85", "s86", "s87", "s88", "s89", "s90", "s91",
        "s92", "s93", "s94", "s95", "s96", "s97", "s98", "s99");
  }
  if (!WARMC && lane == 0) __hip_atomic_store(cntp, call, __ATOMIC_RELAXED, __HIP_MEMORY_SCOPE_AGENT);
  if (dead == 1) {
    asm volatile("s_waitcnt vmcnt(0)" ::: "memory");
    __hip_atomic_store(tmo, call, __ATOMIC_RELAXED, __HIP_MEMORY_SCOPE_AGENT);
    __hip_atomic_store(&ldone[63], 1, __ATOMIC_RELAXED, __HIP_MEMORY_SCOPE_WORKGROUP);
    asm volatile("" ::: "memory");
    if (lane >= k && lane < n) __hip_atomic_fetch_add(&ldone[lane], 1, __ATOMIC_RELAXED, __HIP_MEMORY_SCOPE_WORKGROUP);
    const v4u nan4 = {0x7fc00000u, 0x7fc00000u, 0x7fc00000u, 0x7fc00000u};
    for (; e < n; e += 1 + ALT)
      __builtin_amdgcn_raw_buffer_store_b128(nan4, ro, e_go, (__builtin_amdgcn_readlane(list, e) & 255) << 15, AUX_SYS);
  }
  if (helper && wave < 4 && dead == 2) {
    asm volatile("s_waitcnt vmcnt(0)" ::: "memory");
    if (lane == 0)
      __hip_atomic_store((unsigned*)(rfp + ((idx - 8) * 4 + wave) * FSTR), call, __ATOMIC_RELAXED, __HIP_MEMORY_SCOPE_SYSTEM);
  }
}

extern "C" void kernel_launch(int rank, int nranks, ncclComm_t comm,
                              hipStream_t stream,
                              void* const* d_in, const int* in_sizes, int n_in,
                              void* d_out, int out_size,
                              const int* in_dtypes, int out_dtype,
                              void* const* d_out_all,
                              void* const* d_ws_all, size_t ws_bytes) {
  if (nranks != NR || (unsigned)rank >= (unsigned)NR || n_in < 4 || ws_bytes < WS_NEED ||
      in_sizes[0] != MS * KD || in_sizes[1] != KD * NS || out_size != NR * MS * NS) {
    fprintf(stderr, "kernel_launch: unexpected problem shape\n");
    return;
  }
#define GCL 0x4Du
  const unsigned G = GCL;
  unsigned peers = 0; int k = 0, ci = 0;
  for (int d = 1; d < NR; ++d) {
    const int p = (rank + d) & (NR - 1);
    if (((G >> p) & 1u) == ((G >> rank) & 1u)) { peers |= (unsigned)p << (4 * k++); if (p < rank) ++ci; }
  }
  for (int p = 0; p < NR; ++p)
    if (((G >> p) & 1u) != ((G >> rank) & 1u)) peers |= (unsigned)p << (4 * k++);
  peers |= (unsigned)ci << 28;
  unsigned B = 0x4Du;
  {
    int ndev = 0, dev = -1, bus[NR];
    bool ok = hipGetDeviceCount(&ndev) == hipSuccess && ndev == NR && hipGetDevice(&dev) == hipSuccess && dev == rank;
    for (int p = 0; ok && p < NR; ++p) ok = hipDeviceGetAttribute(&bus[p], hipDeviceAttributePciBusId, p) == hipSuccess;
    unsigned b = 0, seen = 0;
    for (int p = 0; ok && p < NR; ++p) {
      int pos = 0;
      for (int i = 0; i < NR; ++i) pos += bus[i] < bus[p];
      seen |= 1u << pos;
      b |= ((0x4Eu >> pos) & 1u) << p;
    }
    if (ok && seen == 0xffu) B = b;
    (void)hipGetLastError();
  }
  unsigned xmap = 0;
  Recs R;
  {
    int lo = 0, hi = 4;
    for (int role = 0; role < 8; ++role) {
      const unsigned p = role == 3 ? (unsigned)rank : (peers >> (4 * (role < 3 ? role : role - 1))) & 7u;
      const bool nearlo = ((B >> p) & 1u) == ((B >> rank) & 1u);
      const int xc = nearlo ? lo++ : hi++;
      xmap |= (unsigned)role << (4 * xc);
      R.r[xc].ws = (char*)d_ws_all[p];
      R.r[xc].out = role < 4 ? (float*)d_out : (float*)d_out_all[p];
    }
  }
  HIP_CHECK(hipFuncSetAttribute((const void*)ag_gemm_kernel, hipFuncAttributeMaxDynamicSharedMemorySize, LDS_BYTES));
  hipLaunchKernelGGL(ag_gemm_kernel, dim3(GRID), dim3(512), LDS_BYTES, stream,
                     (const int8_t*)d_in[0], (const int8_t*)d_in[1], (char*)d_ws_all[rank], (float*)d_out,
                     (const float*)d_in[2], (const float*)d_in[3], rank, peers, xmap, 0u, R);
  HIP_CHECK(hipGetLastError());
}
```
